```python
import numpy as np

kernel_name = "gpu_gemm_a2a_m4096_k4096_n4096_e5m2e5m2_dequant"


ABSMAX_RTOL = 0.05

COMPUTE_DTYPE = "float32"
EPILOGUE = "dequant"
IN_QUANT = None
OUT_QUANT = None
IN_FP8 = "e5m2e5m2"
IN_DTYPES = {"x": "e5m2", "w_mat": "e5m2",
             "scale_x": "float32", "scale_w": "float32"}
WORLD_SIZE = 8

def _snap_fp8(x, mant_bits, max_abs, min_normal):
    a = np.abs(x.astype(np.float64))
    _m, _e = np.frexp(a)
    q = np.where(a >= min_normal, np.ldexp(1.0, _e - 1 - mant_bits),
                 min_normal * 2.0 ** -mant_bits)
    snapped = np.minimum(np.round(a / q) * q, max_abs)
    return (np.sign(x) * snapped).astype(np.float32)

def setup_inputs():

    x = _snap_fp8(np.random.randn(4096, 4096).astype(np.float32), 2, 57344.0, 2.0 ** -14)
    w_mat = _snap_fp8(np.random.randn(4096, 4096).astype(np.float32), 2, 57344.0, 2.0 ** -14)
    scale_x = (0.02 * (1.0 + np.random.rand(1))).astype(np.float32)
    scale_w = (0.02 * (1.0 + np.random.rand(1))).astype(np.float32)
    return {"x": x, "w_mat": w_mat, "scale_x": scale_x, "scale_w": scale_w}

def reference(x, w_mat, scale_x, scale_w):
    acc = (x @ w_mat).astype(np.float32)
    y = (acc * (scale_x * scale_w)).astype(np.float32)
    return y
```

```cpp
#include <hip/hip_runtime.h>
#include <rccl/rccl.h>
#include <cstdio>

typedef unsigned u32x4 __attribute__((ext_vector_type(4)));
typedef int i32x2 __attribute__((ext_vector_type(2)));
typedef int i32x4 __attribute__((ext_vector_type(4)));
typedef int i32x8 __attribute__((ext_vector_type(8)));
typedef float f32x4 __attribute__((ext_vector_type(4)));
#define LDSP __attribute__((address_space(3)))

constexpr int NR = 8;
constexpr int KD = 4096, ND = 4096;
constexpr int MS = 512, NS_COLS = 512;
constexpr int BM = 64, BN = 128, BK = 128;
constexpr int NKS = KD / BK;
#define NSLOT_ 5
#define MAPPING 0
#define OWAVES 2
#define OPACE 1
#define XPACE 2
constexpr int NSLOT = NSLOT_;
constexpr int NTHR = 512;
constexpr int SLOT_B = (BM + BN) * BK;
constexpr int XROWS = 64;
constexpr int GRID = 256;
constexpr unsigned long long DEADLINE_TICKS = 50000000ull;

constexpr size_t WS_CNT = 0;
constexpr size_t WS_TMO = 4096;
constexpr size_t WS_OFLAG = 8192;
constexpr size_t WS_XFLAG = 12288;
constexpr size_t WS_XSTAGE = 1u << 20;
constexpr size_t XSTAGE_SRC = (size_t)XROWS * KD;
constexpr size_t XSTAGE_PAR = NR * XSTAGE_SRC;
constexpr size_t WS_NEED = WS_XSTAGE + 2 * XSTAGE_PAR;

struct Params {
    const char* x;
    const char* w;
    const float* sx;
    const float* sw;
    char* out[NR];
    char* ws[NR];
    int rank;
    int pad;
};

__device__ __forceinline__ void dma16(unsigned voff, const char* sbase, unsigned lds) {
    asm volatile("s_mov_b32 m0, %2\n\ts_nop 0\n\tglobal_load_lds_dwordx4 %0, %1"
                 :: "v"(voff), "s"(sbase), "s"(lds) : "memory");
}
__device__ __forceinline__ void st_wt(char* p, u32x4 v) {
    asm volatile("global_store_dwordx4 %0, %1, off sc0 sc1\n\ts_nop 1" :: "v"(p), "v"(v) : "memory");
}
__device__ __forceinline__ void wait_flag(unsigned* flag, unsigned call, unsigned* tmo) {
    unsigned long long t0 = __builtin_amdgcn_s_memrealtime();
    while (__hip_atomic_load(flag, __ATOMIC_RELAXED, __HIP_MEMORY_SCOPE_SYSTEM) < call) {
        if (__hip_atomic_load(tmo, __ATOMIC_RELAXED, __HIP_MEMORY_SCOPE_AGENT) == call ||
            __builtin_amdgcn_s_memrealtime() - t0 > DEADLINE_TICKS) {
            __hip_atomic_store(tmo, call, __ATOMIC_RELAXED, __HIP_MEMORY_SCOPE_AGENT);
            break;
        }
        __builtin_amdgcn_s_sleep(1);
    }
}
#define STAMP(i) do { } while (0)

struct Frags { i32x8 x[2], w[4]; };

__device__ __forceinline__ void read_frags(Frags& f, LDSP char* sl, const unsigned (&xrd)[2], const unsigned (&wrd)[4]) {
#pragma unroll
    for (int b = 0; b < 4; ++b) {
        i32x2 t0 = __builtin_amdgcn_ds_read_tr8_b64_v2i32((LDSP i32x2*)(sl + wrd[b]));
        i32x2 t1 = __builtin_amdgcn_ds_read_tr8_b64_v2i32((LDSP i32x2*)(sl + wrd[b] + 8 * BN));
        i32x2 t2 = __builtin_amdgcn_ds_read_tr8_b64_v2i32((LDSP i32x2*)(sl + wrd[b] + 64 * BN));
        i32x2 t3 = __builtin_amdgcn_ds_read_tr8_b64_v2i32((LDSP i32x2*)(sl + wrd[b] + 72 * BN));
        f.w[b] = (i32x8){t0.x, t0.y, t1.x, t1.y, t2.x, t2.y, t3.x, t3.y};
    }
#pragma unroll
    for (int a = 0; a < 2; ++a) {
        i32x4 lo = *(LDSP i32x4*)(sl + xrd[0] + a * 16 * BK);
        i32x4 hi = *(LDSP i32x4*)(sl + xrd[1] + a * 16 * BK);
        f.x[a] = __builtin_shufflevector(lo, hi, 0, 1, 2, 3, 4, 5, 6, 7);
    }
}
__device__ __forceinline__ void mma(f32x4 (&acc)[2][4], const Frags& f) {
#pragma unroll
    for (int b = 0; b < 4; ++b)
#pragma unroll
        for (int a = 0; a < 2; ++a)
            acc[a][b] = __builtin_amdgcn_mfma_scale_f32_16x16x128_f8f6f4(f.w[b], f.x[a], acc[a][b], 1, 1, 0, 0, 0, 0);
}
__device__ __forceinline__ void interleave() {
#pragma unroll
    for (int u = 0; u < 6; ++u) {
        __builtin_amdgcn_sched_group_barrier(0x100, 3, 0);
        __builtin_amdgcn_sched_group_barrier(0x008, 1, 0);
    }
    __builtin_amdgcn_sched_group_barrier(0x100, 2, 0);
    __builtin_amdgcn_sched_group_barrier(0x008, 2, 0);
}
#define CBAR() do { __builtin_amdgcn_sched_barrier(0); asm volatile("s_waitcnt lgkmcnt(0)\n\ts_barrier" ::: "memory"); __builtin_amdgcn_sched_barrier(0); } while (0)

__global__ void __launch_bounds__(NTHR, 2) gemm_a2a(Params p) {
    __shared__ __attribute__((aligned(1024))) char smem[NSLOT * SLOT_B];
    const int tid = threadIdx.x, lane = tid & 63;
    const int wv = __builtin_amdgcn_readfirstlane(tid >> 6);
    const int rank = p.rank;
    char* const myws = p.ws[rank];
    unsigned* const tmo = (unsigned*)(myws + WS_TMO);
    unsigned* const mycnt = (unsigned*)(myws + WS_CNT) + blockIdx.x;

    const int xcd = blockIdx.x & 7, i = blockIdx.x >> 3;
    const int y = xcd >> 1, j = (xcd & 1) * 32 + i;
    int role, bb, rg;
    if (j < 25)      { role = 0; bb = j / 7; rg = j % 7; }
    else if (j < 32) { role = 1; bb = j - 25; rg = 7; }
    else if (j < 56) { role = 0; bb = (j - 7) / 7; rg = (j - 7) % 7; }
    else             { role = 2; bb = 7; rg = j - 56; }
    const int rot = (role == 1 ? bb * 4 + 2 : rg * 4 + bb * 9) & (NKS - 1);
    const int peer = (rank + 1 + bb) & (NR - 1);
    const int dest = role == 0 ? peer : rank;
    const int srcr = role == 1 ? peer : rank;

    unsigned call = 0;
    bool bad = false;
    const char* xbase = p.x + (size_t)rg * BM * KD;
    if (role == 1) {
        const char* src = p.x + (size_t)(MS - XROWS + 16 * y + 2 * wv) * KD + lane * 16;
        call = __hip_atomic_load(mycnt, __ATOMIC_RELAXED, __HIP_MEMORY_SCOPE_SYSTEM) + 1;
        u32x4 v[8];
#pragma unroll
        for (int u = 0; u < 8; ++u) v[u] = __builtin_nontemporal_load((const u32x4*)(src + u * 1024));
        call = __builtin_amdgcn_readfirstlane(call);
        char* dst = p.ws[peer] + WS_XSTAGE + (call & 1) * XSTAGE_PAR + rank * XSTAGE_SRC + (size_t)(16 * y + 2 * wv) * KD + lane * 16;
#pragma unroll
        for (int u = 0; u < 8; ++u) {
            st_wt(dst + u * 1024, v[u]);
            if (XPACE > 0) asm volatile("s_waitcnt vmcnt(%0)" :: "i"(XPACE > 0 ? XPACE - 1 : 0) : "memory");
        }
        asm volatile("s_waitcnt vmcnt(0)" ::: "memory");
        __syncthreads();
        if (tid == 0)
            __hip_atomic_store((unsigned*)(p.ws[peer] + WS_XFLAG) + rank * 32 + y, call, __ATOMIC_RELAXED, __HIP_MEMORY_SCOPE_SYSTEM);
        STAMP(4);
        if (tid < 4) {
            wait_flag((unsigned*)(myws + WS_XFLAG) + peer * 32 + tid, call, tmo);
            __builtin_amdgcn_fence(__ATOMIC_ACQUIRE, "");
            asm volatile("s_waitcnt vmcnt(0)" ::: "memory");
        }
        __syncthreads();
        bad = __hip_atomic_load(tmo, __ATOMIC_RELAXED, __HIP_MEMORY_SCOPE_AGENT) == call;
        xbase = myws + WS_XSTAGE + (call & 1) * XSTAGE_PAR + peer * XSTAGE_SRC;
        STAMP(5);
    }
    const char* wbase = p.w + dest * NS_COLS + y * BN;
    const int g = lane >> 4, li = lane & 15;
    const int grow0 = srcr * MS + rg * BM;
    char* const obase = p.out[dest] + ((size_t)grow0 * NS_COLS + y * BN) * 4;

    if (wv >= 4) {
        const int lw = wv - 4;
        const unsigned lds0 = (unsigned)(size_t)(LDSP char*)smem;
        unsigned xoff[2], woff[4];
        const int lr = lane >> 3, sl = lane & 7;
#pragma unroll
        for (int u = 0; u < 2; ++u) {
            int row = 8 * (2 * lw + u) + lr;
            xoff[u] = row * KD + ((sl ^ ((row >> 1) & 7)) << 4);
        }
#pragma unroll
        for (int u = 0; u < 4; ++u) {
            int r = 8 * (4 * lw + u) + lr;
            woff[u] = r * ND + ((sl ^ (((r >> 1) & 3) | ((r >> 2) & 4))) << 4);
        }
        unsigned slot = 0;
        auto issue = [&](int s) {
            const int kk = (s + rot) & (NKS - 1);
            const char* xb = xbase + kk * BK;
            const char* wb = wbase + (size_t)kk * BK * ND;
            const unsigned l = lds0 + slot;
#pragma unroll
            for (int u = 0; u < 4; ++u) dma16(woff[u], wb, l + BM * BK + (4 * lw + u) * 1024);
#pragma unroll
            for (int u = 0; u < 2; ++u) dma16(xoff[u], xb, l + (2 * lw + u) * 1024);
            slot = slot + SLOT_B == NSLOT * SLOT_B ? 0 : slot + SLOT_B;
        };
#pragma unroll 1
        for (int s = 0; s < NSLOT - 1; ++s) issue(s);
#pragma unroll 1
        for (int s = 0; s < NKS - (NSLOT - 1); ++s) {
            asm volatile("s_waitcnt vmcnt(%0)\n\ts_barrier" :: "i"(6 * (NSLOT - 2)) : "memory");
            issue(s + NSLOT - 1);
        }
#pragma unroll
        for (int s = NSLOT - 2; s >= 0; --s)
            asm volatile("s_waitcnt vmcnt(%0)\n\ts_barrier" :: "i"(6 * s) : "memory");
        asm volatile("s_barrier" ::: "memory");
        asm volatile("s_barrier" ::: "memory");
    } else {
        const int wm = wv >> 1, wn = wv & 1;
        unsigned xrd[2], wrd[4];
        const int fx = (li >> 1) & 7;
        xrd[0] = (32 * wm + li) * BK + ((g ^ fx) << 4);
        xrd[1] = (32 * wm + li) * BK + (((g + 4) ^ fx) << 4);
        const int q = li >> 1, pp = li & 1;
        const int fw = ((q >> 1) & 3) | ((g & 1) << 2);
#pragma unroll
        for (int b = 0; b < 4; ++b) wrd[b] = BM * BK + (16 * g + q) * BN + (((4 * wn + b) ^ fw) << 4) + 8 * pp;

        f32x4 acc[2][4];
#pragma unroll
        for (int a = 0; a < 2; ++a)
#pragma unroll
            for (int b = 0; b < 4; ++b) acc[a][b] = (f32x4){0.f, 0.f, 0.f, 0.f};
        float scale = p.sx[0] * p.sw[0];
        if (bad) scale = __builtin_nanf("");
        unsigned cnt = __hip_atomic_load(mycnt, __ATOMIC_RELAXED, __HIP_MEMORY_SCOPE_SYSTEM);

        Frags fa, fb;
        unsigned slot = 0;
        auto next = [&]() { LDSP char* r = (LDSP char*)smem + slot; slot = slot + SLOT_B == NSLOT * SLOT_B ? 0 : slot + SLOT_B; return r; };
        CBAR();
        read_frags(fa, next(), xrd, wrd);
#pragma unroll 1
        for (int s = 1; s < NKS - 1; s += 2) {
            CBAR();
            read_frags(fb, next(), xrd, wrd);
            mma(acc, fa);
            interleave();
            CBAR();
            read_frags(fa, next(), xrd, wrd);
            mma(acc, fb);
            interleave();
        }
        CBAR();
        read_frags(fb, next(), xrd, wrd);
        mma(acc, fa);
        interleave();
        CBAR();
        mma(acc, fb);
        STAMP(1);
#pragma unroll
        for (int a = 0; a < 2; ++a)
#pragma unroll
            for (int b = 0; b < 4; ++b) {
                const int row = 32 * wm + 16 * a + li;
                *(LDSP f32x4*)((LDSP char*)smem + row * 512 + (((16 * wn + 4 * b + g) ^ (row & 7)) << 4)) = acc[a][b] * scale;
            }
        call = cnt + 1;
        CBAR();
    }

    if (wv < OWAVES) {
        u32x4 v[32 / OWAVES];
        const int c = lane & 31;
#pragma unroll
        for (int u = 0; u < 32 / OWAVES; ++u) {
            const int row = 2 * (u * OWAVES + wv) + (lane >> 5);
            v[u] = *(LDSP u32x4*)((LDSP char*)smem + row * 512 + ((c ^ (row & 7)) << 4));
        }
#pragma unroll
        for (int u = 0; u < 32 / OWAVES; ++u) {
            const int row = 2 * (u * OWAVES + wv) + (lane >> 5);
            st_wt(obase + (size_t)row * NS_COLS * 4 + c * 16, v[u]);
            if (OPACE > 0) asm volatile("s_waitcnt vmcnt(%0)" :: "i"(OPACE > 0 ? OPACE - 1 : 0) : "memory");
        }
    }
    STAMP(2);
    asm volatile("s_waitcnt vmcnt(0)" ::: "memory");
    __syncthreads();
    if (tid == 0) {
        if (role == 0)
            __hip_atomic_store((unsigned*)(p.ws[dest] + WS_OFLAG) + rank * 32 + y * 7 + rg, call, __ATOMIC_RELAXED, __HIP_MEMORY_SCOPE_SYSTEM);
        __hip_atomic_store(mycnt, call, __ATOMIC_RELAXED, __HIP_MEMORY_SCOPE_SYSTEM);
    }
    STAMP(3);
    if (blockIdx.x == GRID - 1) {
        const int s = tid >> 5, f = tid & 31;
        if (s < NR && s != rank && f < 28) wait_flag((unsigned*)(myws + WS_OFLAG) + s * 32 + f, call, tmo);
        __syncthreads();
        if (tid == 0 && __hip_atomic_load(tmo, __ATOMIC_RELAXED, __HIP_MEMORY_SCOPE_AGENT) == call)
            *(float*)obase = __builtin_nanf("");
        STAMP(6);
    }
}

extern "C" void kernel_launch(int rank, int nranks, ncclComm_t comm,
                              hipStream_t stream,
                              void* const* d_in, const int* in_sizes, int n_in,
                              void* d_out, int out_size,
                              const int* in_dtypes, int out_dtype,
                              void* const* d_out_all,
                              void* const* d_ws_all, size_t ws_bytes) {
    if (nranks != NR || (unsigned)rank >= (unsigned)NR || n_in < 4 || ws_bytes < WS_NEED ||
        in_sizes[0] != MS * KD || in_sizes[1] != KD * ND || out_size != NR * MS * NS_COLS) {
        fprintf(stderr, "kernel_launch: unexpected problem shape\n");
        return;
    }
    Params p;
    p.x = (const char*)d_in[0];
    p.w = (const char*)d_in[1];
    p.sx = (const float*)d_in[2];
    p.sw = (const float*)d_in[3];
    for (int r = 0; r < NR; ++r) { p.out[r] = (char*)d_out_all[r]; p.ws[r] = (char*)d_ws_all[r]; }
    p.rank = rank;
    p.pad = 0;
    gemm_a2a<<<GRID, NTHR, 0, stream>>>(p);
}
```

```cpp
#include <hip/hip_runtime.h>
#include <rccl/rccl.h>
#include <cstdio>

typedef unsigned u32x4 __attribute__((ext_vector_type(4)));
typedef int i32x2 __attribute__((ext_vector_type(2)));
typedef int i32x4 __attribute__((ext_vector_type(4)));
typedef int i32x8 __attribute__((ext_vector_type(8)));
typedef float f32x4 __attribute__((ext_vector_type(4)));
typedef unsigned long long u64;
#define LDSP __attribute__((address_space(3)))

constexpr int NR = 8;
constexpr int KD = 4096, ND = 4096;
constexpr int MS = 512, NS_COLS = 512;
constexpr int BN = 128, BK = 128;
constexpr int NKS = KD / BK;
#define NSLOT_ 5
#define PF3T 250
#define NPF_ 10
#define PF4T 500
constexpr int NPF = NPF_;
#define XPACE 3
#define NXE 6
#define XPF 0
#define XLEAD 120
#define POLLSLEEP 2
#define MAXESC 16
constexpr int NSLOT = NSLOT_;
constexpr int NTHR = 512;
constexpr int SLOT_B = (64 + BN) * BK;
constexpr int XROWS = 64;
constexpr int NXPART = 4;
constexpr int GRID = 256;
constexpr int NTILE = 32;
constexpr u64 DEADLINE_TICKS = 50000000ull;

constexpr size_t WS_CNT = 0;
constexpr size_t WS_TMO = 8192;
constexpr size_t WS_XFLAG = 12288;
constexpr size_t WS_UFLAG = 16384;
constexpr size_t UFLAG_SRC = NTILE * 4 * 2 * 8, UFLAG_PAR = NR * UFLAG_SRC;
constexpr size_t WS_XSTAGE = 1u << 20;
constexpr size_t XSTAGE_SRC = (size_t)XROWS * KD, XSTAGE_PAR = NR * XSTAGE_SRC;
constexpr size_t WS_USTAGE = 8u << 20;
constexpr int UTILE = 4 * 2 * 3200;
constexpr size_t USTAGE_SRC = 1u << 20, USTAGE_PAR = NR * USTAGE_SRC;
static_assert(NTILE * UTILE <= USTAGE_SRC, "staging");
constexpr size_t WS_NEED = WS_USTAGE + 2 * USTAGE_PAR;
static_assert((NKS - 1) % NSLOT < 2 && NSLOT >= 4, "the receivers' patches sit in ring slots 2 and 3");
static_assert(WS_UFLAG + 2 * UFLAG_PAR <= 65536 && WS_XSTAGE + 2 * XSTAGE_PAR <= WS_USTAGE, "workspace layout");

enum { PRE_NONE = 0, PRE_XSEND = 1, PRE_DELAY = 2 };
enum { T_NONE = 0, T_REMOTE = 1, T_OWN = 2, T_PEER = 3 };
struct Prog {
    unsigned pre;
    unsigned t[2];
};
static unsigned pack_tile(int rank, int kind, int mt, int row16, int bb, int y, int rot, int flag, int npk) {
    const unsigned cb = (kind == T_REMOTE ? (rank + 1 + bb) % NR : rank) * 4 + y;
    return kind | (mt - 1) << 2 | row16 << 3 | bb << 8 | y << 11 | rot << 13 | flag << 20 | npk << 25 | cb << 27;
}
struct TileInfo { int kind, mt, row0, y, bb, peer, rot, flag, npk; };
__device__ __forceinline__ TileInfo decode(unsigned t, int rank) {
    TileInfo d;
    d.kind = t & 3; d.mt = ((t >> 2) & 1) + 1; d.row0 = ((t >> 3) & 31) * 16; d.bb = (t >> 8) & 7; d.y = (t >> 11) & 3;
    d.peer = (rank + 1 + d.bb) & (NR - 1); d.rot = (t >> 13) & 31; d.flag = (t >> 20) & 31;
    d.npk = (t >> 25) & 3;
    return d;
}

struct Params {
    const char* x;
    const char* w;
    const float* sx;
    const float* sw;
    char* out[NR];
    char* ws[NR];
    int rank;
    int pad;
    Prog prog[GRID];
};

__device__ __forceinline__ void touch(unsigned lds, unsigned voff, const char* sbase) {
    asm volatile("s_mov_b32 m0, %2\n\ts_nop 0\n\tglobal_load_lds_dword %0, %1" :: "v"(voff), "s"(sbase), "s"(lds) : "memory");
}
__device__ __forceinline__ void st_wt(char* p, u32x4 v) {
    asm volatile("global_store_dwordx4 %0, %1, off sc0 sc1\n\ts_nop 1" :: "v"(p), "v"(v) : "memory");
}
__device__ __forceinline__ void st_wt1(char* p, float v) {
    asm volatile("global_store_dword %0, %1, off sc0 sc1" :: "v"(p), "v"(v) : "memory");
}
__device__ __forceinline__ unsigned next_call(unsigned* cnt) {
    const unsigned c = __builtin_amdgcn_readfirstlane(__hip_atomic_load(cnt, __ATOMIC_RELAXED, __HIP_MEMORY_SCOPE_SYSTEM)) + 1;
    if ((threadIdx.x & 63) == 0) __hip_atomic_store(cnt, c, __ATOMIC_RELAXED, __HIP_MEMORY_SCOPE_SYSTEM);
    return c;
}
__device__ __forceinline__ void wait_flag(unsigned* flag, unsigned call, unsigned* tmo) {
    u64 t0 = __builtin_amdgcn_s_memrealtime();
    while (__hip_atomic_load(flag, __ATOMIC_RELAXED, __HIP_MEMORY_SCOPE_SYSTEM) < call) {
        if (__hip_atomic_load(tmo, __ATOMIC_RELAXED, __HIP_MEMORY_SCOPE_AGENT) == call ||
            __builtin_amdgcn_s_memrealtime() - t0 > DEADLINE_TICKS) {
            __hip_atomic_store(tmo, call, __ATOMIC_RELAXED, __HIP_MEMORY_SCOPE_AGENT);
            break;
        }
        __builtin_amdgcn_s_sleep(2);
    }
}
__device__ __forceinline__ bool wait_uflag(u64* flag, unsigned call, unsigned* tmo, unsigned& hdr) {
    const u64 t0 = __builtin_amdgcn_s_memrealtime();
    for (unsigned it = 1;; ++it) {
        const u64 f = __hip_atomic_load(flag, __ATOMIC_RELAXED, __HIP_MEMORY_SCOPE_SYSTEM);
        hdr = __builtin_amdgcn_readfirstlane((unsigned)f);
        if (__builtin_amdgcn_readfirstlane((unsigned)(f >> 32)) >= call) return true;
        if ((it & 63) == 0) {
            if (__hip_atomic_load(tmo, __ATOMIC_RELAXED, __HIP_MEMORY_SCOPE_AGENT) == call ||
                __builtin_amdgcn_s_memrealtime() - t0 > DEADLINE_TICKS) {
                __hip_atomic_store(tmo, call, __ATOMIC_RELAXED, __HIP_MEMORY_SCOPE_AGENT);
                return false;
            }
        }
        __builtin_amdgcn_s_sleep(POLLSLEEP);
    }
}
#define STAMP(i) do { } while (0)
#define LSTAMP(i, v) do { } while (0)
#define PROF(i) do { } while (0)
#define PROFOUT(base, n) do { } while (0)

template <int MT> struct Frags { i32x8 x[2], w[2 * MT]; };

template <int MT>
__device__ __forceinline__ void read_frags(Frags<MT>& f, LDSP char* sl, const unsigned (&xrd)[2], const unsigned (&wrd)[2 * MT]) {
#pragma unroll
    for (int b = 0; b < 2 * MT; ++b) {
        i32x2 t0 = __builtin_amdgcn_ds_read_tr8_b64_v2i32((LDSP i32x2*)(sl + wrd[b]));
        i32x2 t1 = __builtin_amdgcn_ds_read_tr8_b64_v2i32((LDSP i32x2*)(sl + wrd[b] + 8 * BN));
        i32x2 t2 = __builtin_amdgcn_ds_read_tr8_b64_v2i32((LDSP i32x2*)(sl + wrd[b] + 64 * BN));
        i32x2 t3 = __builtin_amdgcn_ds_read_tr8_b64_v2i32((LDSP i32x2*)(sl + wrd[b] + 72 * BN));
        f.w[b] = (i32x8){t0.x, t0.y, t1.x, t1.y, t2.x, t2.y, t3.x, t3.y};
        if (b == 0) {
#pragma unroll
            for (int a = 0; a < 2; ++a) {
                i32x4 lo = *(LDSP i32x4*)(sl + xrd[0] + a * 4096);
                i32x4 hi = *(LDSP i32x4*)(sl + xrd[1] + a * 4096);
                f.x[a] = __builtin_shufflevector(lo, hi, 0, 1, 2, 3, 4, 5, 6, 7);
            }
        }
    }
}
template <int MT>
__device__ __forceinline__ void mma(f32x4 (&acc)[2][2 * MT], const Frags<MT>& f) {
#pragma unroll
    for (int b = 0; b < 2 * MT; ++b)
#pragma unroll
        for (int a = 0; a < 2; ++a)
            acc[a][b] = __builtin_amdgcn_mfma_scale_f32_16x16x128_f8f6f4(f.w[b], f.x[a], acc[a][b], 1, 1, 0, 0, 0, 0);
}
template <int MT>
__device__ __forceinline__ void interleave() {
    if (MT == 2) {
#pragma unroll
        for (int u = 0; u < 6; ++u) {
            __builtin_amdgcn_sched_group_barrier(0x100, 3, 0);
            __builtin_amdgcn_sched_group_barrier(0x008, 1, 0);
        }
        __builtin_amdgcn_sched_group_barrier(0x100, 2, 0);
        __builtin_amdgcn_sched_group_barrier(0x008, 2, 0);
    } else {
#pragma unroll
        for (int u = 0; u < 4; ++u) {
            __builtin_amdgcn_sched_group_barrier(0x100, 3, 0);
            __builtin_amdgcn_sched_group_barrier(0x008, 1, 0);
        }
    }
}
#define CBAR() do { __builtin_amdgcn_sched_barrier(0); asm volatile("s_waitcnt lgkmcnt(0)\n\ts_barrier" ::: "memory"); __builtin_amdgcn_sched_barrier(0); } while (0)
constexpr int AHEAD = NSLOT - 1;

#define LD_ISSUE(XTRA) \
    "s_mov_b32 m0, %[slot]\n\t" \
    "s_lshr_b32 %[sox], %[sow], 12\n\t" \
    "buffer_load_dwordx4 %[w0], %[rw], %[sow] offen lds\n\t" \
    "buffer_load_dwordx4 %[w1], %[rw], %[sow] offen offset:1024 lds\n\t" \
    "buffer_load_dwordx4 %[w2], %[rw], %[sow] offen offset:2048 lds\n\t" \
    "buffer_load_dwordx4 %[w3], %[rw], %[sow] offen offset:3072 lds\n\t" \
    "s_add_u32 m0, %[slot], %[ldsx]\n\t" \
    "s_add_u32 %[slot], %[slot], %[slotb]\n\t" \
    "buffer_load_dwordx4 %[x0], %[rx], %[sox] offen lds\n\t" \
    XTRA \
    "s_cmp_eq_u32 %[slot], %[send]\n\t" \
    "s_cselect_b32 %[slot], %[sbeg], %[slot]\n\t" \
    "s_add_u32 %[sow], %[sow], 0x80000\n\t" \
    "s_and_b32 %[sow], %[sow], 0xffffff\n\t"
struct Stream { unsigned w0, w1, w2, w3, x0, x1, ldsw, ldsx; };
#define LD_OPS \
    [w0] "v"(k.w0), [w1] "v"(k.w1), [w2] "v"(k.w2), [w3] "v"(k.w3), [x0] "v"(k.x0), [x1] "v"(k.x1), \
    [rw] "s"(rw), [rx] "s"(rx), [ldsx] "s"(k.ldsx), [sbeg] "s"(k.ldsw), [send] "s"(k.ldsw + NSLOT * SLOT_B), [slotb] "i"(SLOT_B)
#define LD_X1 "buffer_load_dwordx4 %[x1], %[rx], %[sox] offen offset:1024 lds\n\t"
__device__ __forceinline__ u32x4 rsrc(const char* base, unsigned bytes) {
    return (u32x4){(unsigned)(u64)base, (unsigned)((u64)base >> 32), bytes, 0x00020000u};
}
__device__ __forceinline__ void stream_start(const Stream& k, const char* wbase, const char* xbase, unsigned rot, unsigned big) {
    const u32x4 rw = rsrc(wbase, KD * ND), rx = rsrc(xbase, 64 * KD);
    unsigned slot = k.ldsw, sow = rot * (BK * ND), sox, n;
    asm volatile("s_nop 1\n\ts_mov_b32 %[n], %[ahead]\n"
                 "1:\n\t" LD_ISSUE("s_cmp_eq_u32 %[big], 0\n\ts_cbranch_scc1 3f\n\t" LD_X1 "3:\n\t")
                 "s_sub_u32 %[n], %[n], 1\n\t"
                 "s_cmp_lg_u32 %[n], 0\n\t"
                 "s_cbranch_scc1 1b"
                 : [slot] "+&s"(slot), [sow] "+&s"(sow), [sox] "=&s"(sox), [n] "=&s"(n)
                 : LD_OPS, [ahead] "i"(AHEAD), [big] "s"(big) : "memory", "scc");
}
#define LD_MAIN(XTRA) \
    "s_mov_b32 %[n], %[nmain]\n" \
    "2:\n\t" \
    "s_waitcnt vmcnt(%[c2])\n\t" \
    "s_barrier\n\t" LD_ISSUE(XTRA) \
    "s_sub_u32 %[n], %[n], 1\n\t" \
    "s_cmp_lg_u32 %[n], 0\n\t" \
    "s_cbranch_scc1 2b"
template <int MT>
__device__ __forceinline__ void stream_rest(const Stream& k, const char* wbase, const char* xbase, unsigned rot) {
    const u32x4 rw = rsrc(wbase, KD * ND), rx = rsrc(xbase, 64 * KD);
    unsigned slot = k.ldsw + AHEAD * SLOT_B, sow = ((rot + AHEAD) & (NKS - 1)) * (BK * ND), sox, n;
    constexpr int PER = 4 + MT;
    if (MT == 2)
        asm volatile(LD_MAIN(LD_X1) : [slot] "+&s"(slot), [sow] "+&s"(sow), [sox] "=&s"(sox), [n] "=&s"(n)
                     : LD_OPS, [nmain] "i"(NKS - AHEAD), [c2] "i"(PER * (AHEAD - 1)) : "memory", "scc");
    else
        asm volatile(LD_MAIN("") : [slot] "+&s"(slot), [sow] "+&s"(sow), [sox] "=&s"(sox), [n] "=&s"(n)
                     : LD_OPS, [nmain] "i"(NKS - AHEAD), [c2] "i"(PER * (AHEAD - 1)) : "memory", "scc");
#pragma unroll
    for (int s = AHEAD - 1; s >= 0; --s)
        asm volatile("s_waitcnt vmcnt(%0)\n\ts_barrier" :: "i"(PER * s) : "memory");
}

__device__ __forceinline__ float wave_sum(float v) {
    v += __builtin_bit_cast(float, __builtin_amdgcn_update_dpp(0, __builtin_bit_cast(int, v), 0x111, 0xf, 0xf, true));
    v += __builtin_bit_cast(float, __builtin_amdgcn_update_dpp(0, __builtin_bit_cast(int, v), 0x112, 0xf, 0xf, true));
    v += __builtin_bit_cast(float, __builtin_amdgcn_update_dpp(0, __builtin_bit_cast(int, v), 0x114, 0xf, 0xf, true));
    v += __builtin_bit_cast(float, __builtin_amdgcn_update_dpp(0, __builtin_bit_cast(int, v), 0x118, 0xf, 0xf, true));
    const int i = __builtin_bit_cast(int, v);
    return (__builtin_bit_cast(float, __builtin_amdgcn_readlane(i, 15)) + __builtin_bit_cast(float, __builtin_amdgcn_readlane(i, 31))) +
           (__builtin_bit_cast(float, __builtin_amdgcn_readlane(i, 47)) + __builtin_bit_cast(float, __builtin_amdgcn_readlane(i, 63)));
}
template <int MT, class Hook>
__device__ __forceinline__ void compute(LDSP char* smem, int wv, int lane, f32x4 (&out)[8], Hook hook) {
    const int g = lane >> 4, li = lane & 15;
    const int wm = MT == 2 ? wv >> 1 : 0, wc = MT == 2 ? 4 * (wv & 1) : 2 * wv;
    unsigned xrd[2], wrd[2 * MT];
    const int fx = (li >> 1) & 7;
    xrd[0] = BN * BK + (li >> 3) * 2048 + wm * 1024 + (li & 7) * BK + ((g ^ fx) << 4);
    xrd[1] = xrd[0] ^ 64;
    const int q = li >> 1, pp = li & 1;
    const int fw = ((q >> 1) & 3) | ((g & 1) << 2);
#pragma unroll
    for (int b = 0; b < 2 * MT; ++b) wrd[b] = (16 * g + q) * BN + (((wc + b) ^ fw) << 4) + 8 * pp;
    f32x4 acc[2][2 * MT];
#pragma unroll
    for (int a = 0; a < 2; ++a)
#pragma unroll
        for (int b = 0; b < 2 * MT; ++b) acc[a][b] = (f32x4){0.f, 0.f, 0.f, 0.f};
    Frags<MT> fa, fb;
    unsigned slot = 0;
    auto next = [&]() { LDSP char* r = smem + slot; slot = slot + SLOT_B == NSLOT * SLOT_B ? 0 : slot + SLOT_B; return r; };
    CBAR();
    read_frags<MT>(fa, next(), xrd, wrd);
#pragma unroll 1
    for (int s = 1; s < NKS - 1; s += 2) {
        CBAR();
        read_frags<MT>(fb, next(), xrd, wrd);
        mma<MT>(acc, fa);
        interleave<MT>();
        CBAR();
        read_frags<MT>(fa, next(), xrd, wrd);
        mma<MT>(acc, fb);
        interleave<MT>();
    }
    __builtin_amdgcn_sched_barrier(0);
    {
        const float a0 = acc[0][0][0], a1 = acc[0][0][1], a2 = acc[0][0][2], a3 = acc[0][0][3];
        hook(wave_sum(__builtin_fabsf(a0) + __builtin_fabsf(a1) + __builtin_fabsf(a2) + __builtin_fabsf(a3)) * (1.f / 64));
    }
    CBAR();
    read_frags<MT>(fb, next(), xrd, wrd);
    mma<MT>(acc, fa);
    interleave<MT>();
    __builtin_amdgcn_sched_barrier(0);
#pragma unroll
    for (int a = 0; a < 2; ++a)
#pragma unroll
        for (int b = 0; b < 2 * MT; ++b)
            out[2 * MT * a + b] = __builtin_amdgcn_mfma_scale_f32_16x16x128_f8f6f4(fb.w[b], fb.x[a], acc[a][b], 1, 1, 0, 0, 0, 0);
}
__device__ __forceinline__ unsigned frag_slot(int mt, int lane, int f) {
    const int g = lane >> 4, li = lane & 15;
    return mt == 2 ? li * 256 + (((4 * f + g) ^ (li & 7)) << 4) : (16 * (f >> 1) + li) * 128 + (((4 * (f & 1) + g) ^ (li & 7)) << 4);
}
__device__ __forceinline__ unsigned piece_slot(int mt, int lane, int u) {
    return u * 1024 + ((lane ^ (mt == 2 ? (4 * u + (lane >> 4)) & 7 : lane >> 3)) << 4);
}
__device__ __forceinline__ void piece_geom(int mt, int wq, int lane, unsigned& off, unsigned& step) {
    const int r0 = mt == 2 ? lane >> 4 : lane >> 3, c = mt == 2 ? lane & 15 : lane & 7;
    off = (mt == 2 ? (32 * (wq >> 1) + r0) * NS_COLS * 4 + (wq & 1) * 256 : r0 * NS_COLS * 4 + wq * 128) + c * 16;
    step = (mt == 2 ? 4 : 8) * NS_COLS * 4;
}

__global__ void __launch_bounds__(NTHR, 2) gemm_a2a(Params p) {
    __shared__ __attribute__((aligned(1024))) char smem_[NSLOT * SLOT_B + 16384 + 2048];
    LDSP char* smem = (LDSP char*)smem_;
    const int tid = threadIdx.x, lane = tid & 63;
    const int wv = __builtin_amdgcn_readfirstlane(tid >> 6);
    const u64 tstart = __builtin_amdgcn_s_memrealtime();
    const int rank = p.rank;
    char* const myws = p.ws[NR - 1];
    const char* const px = p.x;
    const char* const pw = p.w;
    const unsigned pre = p.prog[blockIdx.x].pre;
    const unsigned tile0 = p.prog[blockIdx.x].t[0], tile1 = p.prog[blockIdx.x].t[1];
    Stream ks;
    {
        const int lw = wv & 3, lr = lane >> 3, sl = lane & 7;
        ks.ldsw = (unsigned)(size_t)smem + lw * 4096;
        ks.ldsx = BN * BK - lw * 2048;
        ks.w0 = (32 * lw + lr) * ND + ((sl ^ (lr >> 1)) << 4);
        ks.w1 = ks.w0 + 8 * ND - 1024; ks.w2 = (ks.w0 ^ 64) + 16 * ND - 2048; ks.w3 = ks.w2 + 8 * ND - 1024;
        ks.x0 = (8 * lw + lr) * KD + ((sl ^ ((lr >> 1) | ((lw & 1) << 2))) << 4);
        ks.x1 = ks.x0 + 32 * KD - 1024;
        asm volatile("" : "+v"(ks.w0), "+v"(ks.w1), "+v"(ks.w2), "+v"(ks.w3), "+v"(ks.x0), "+v"(ks.x1));
        asm volatile("" : "+s"(ks.ldsw), "+s"(ks.ldsx));
    }
    __builtin_amdgcn_sched_barrier(0);
    const bool early = wv >= 4 && (pre & 3) == PRE_NONE;
    if (early) stream_start(ks, pw + (tile0 >> 27) * BN, px + ((tile0 >> 3) & 31) * 16 * KD, 0, tile0 & 4);
    __builtin_amdgcn_sched_barrier(0);
    unsigned* const tmo = (unsigned*)(myws + WS_TMO);
    unsigned* const mycnt = (unsigned*)(myws + WS_CNT) + blockIdx.x * 8 + wv;
    unsigned call = 0;

    if (__builtin_expect((pre & 3) != PRE_NONE, 0)) {
        if ((pre & 3) == PRE_XSEND) {
            const int bb = (pre >> 15) & 7, part = (pre >> 13) & 3;
            const char* src = px + (size_t)(MS - XROWS + 16 * part + 2 * wv) * KD + lane * 16;
            const unsigned cnt = __hip_atomic_load(mycnt, __ATOMIC_RELAXED, __HIP_MEMORY_SCOPE_SYSTEM);
            u32x4 v[8];
#pragma unroll
            for (int u = 0; u < 8; ++u) v[u] = *(const u32x4*)(src + u * 1024);
            char* const pws = p.ws[bb];
            call = __builtin_amdgcn_readfirstlane(cnt) + 1;
            char* dst = pws + WS_XSTAGE + (call & 1) * XSTAGE_PAR + rank * XSTAGE_SRC + (size_t)(16 * part + 2 * wv) * KD + lane * 16;
#pragma unroll
            for (int u = 0; u < 8; ++u) {
                st_wt(dst + u * 1024, v[u]);
                if (XPACE > 0 && u >= XPACE - 1) asm volatile("s_waitcnt vmcnt(%0)" :: "i"(XPACE > 0 ? XPACE - 1 : 0) : "memory");
            }
            asm volatile("s_waitcnt vmcnt(0)" ::: "memory");
            __syncthreads();
            if (tid == 0)
                __hip_atomic_store((unsigned*)(pws + WS_XFLAG) + rank * 32 + part, call, __ATOMIC_RELAXED, __HIP_MEMORY_SCOPE_SYSTEM);
            if (lane == 0) __hip_atomic_store(mycnt, call, __ATOMIC_RELAXED, __HIP_MEMORY_SCOPE_SYSTEM);
            STAMP(4);
            {
                const unsigned dmy = (unsigned)(size_t)smem + NSLOT * SLOT_B + 16384, id4 = wv * 4 + part;
                const char* wo = pw + rank * NS_COLS + (bb >> 1) * BN;
                touch(dmy, (id4 * 64 + lane) * ND, wo);
                touch(dmy, (2048 + id4 * 64 + lane) * ND, wo);
            }
            while (__builtin_amdgcn_s_memrealtime() - tstart < ((pre >> 2) & 2047)) __builtin_amdgcn_s_sleep(2);
        } else {
            const unsigned ticks = (pre >> 2) & 2047;
            const int job = (pre >> 13) & 31;
            const unsigned dmy = (unsigned)(size_t)smem + NSLOT * SLOT_B + 16384;
            if (job & 15) {
                const int bb = (tile0 >> 8) & 7;
                const char* wp = pw + ((rank + 1 + bb) & (NR - 1)) * NS_COLS;
                const int id = wv * NPF + job - 1;
#pragma unroll 1
                for (int idx = id; idx < 352; idx += 8 * NPF) {
                    const int st = idx / 11, part = idx % 11;
                    if (part < 4) touch(dmy, (st * BK + 32 * part + (lane >> 1)) * ND + (lane & 1) * BN, wp);
                    else          touch(dmy, (64 * (part - 4) + lane) * KD + st * BK, px);
                }
                while (__builtin_amdgcn_s_memrealtime() - tstart < PF3T) __builtin_amdgcn_s_sleep(2);
                if (id < 64) touch(dmy, (id * 64 + lane) * ND + 2 * BN, wp);
                if (job > NPF - 4) {
                    while (__builtin_amdgcn_s_memrealtime() - tstart < PF4T) __builtin_amdgcn_s_sleep(2);
                    const int id4 = wv * 4 + job - (NPF - 3);
                    touch(dmy, (id4 * 64 + lane) * ND + 3 * BN, wp);
                    touch(dmy, (2048 + id4 * 64 + lane) * ND + 3 * BN, wp);
                }
            }
            while (__builtin_amdgcn_s_memrealtime() - tstart < ticks) __builtin_amdgcn_s_sleep(2);
            if (job == 16)
                touch(dmy, ((wv * 32 + (blockIdx.x >> 3)) * 16 + (lane >> 2)) * ND + (lane & 3) * BN, pw + rank * NS_COLS);
        }
    }

    if (wv >= 4) {
        const int lw = wv - 4;
#pragma unroll 1
        for (int ti = 0; ti < 2; ++ti) {
            const TileInfo d = decode(ti ? tile1 : tile0, rank);
            if (d.kind == T_NONE) break;
            const char* xbase = px + (size_t)d.row0 * KD;
            const char* wbase = pw + ((ti ? tile1 : tile0) >> 27) * BN;
            if (ti) asm volatile("s_barrier" ::: "memory");
            if (__builtin_expect(d.kind == T_PEER, 0)) {
                if (!call) call = next_call(mycnt);
                xbase = myws + WS_XSTAGE + (call & 1) * XSTAGE_PAR + d.peer * XSTAGE_SRC + (size_t)(d.row0 - (MS - XROWS)) * KD;
                asm volatile("s_barrier" ::: "memory");
            }
            if (ti || !early) stream_start(ks, wbase, xbase, d.rot, d.mt - 1);
            if (d.mt == 2) stream_rest<2>(ks, wbase, xbase, d.rot);
            else           stream_rest<1>(ks, wbase, xbase, d.rot);
        }
        if (!call) call = next_call(mycnt);
    } else {
        LDSP char* ep = smem + NSLOT * SLOT_B + wv * 4096;
        LDSP char* pp = smem + NSLOT * SLOT_B + 16384 + 256 + wv * 256;
        volatile LDSP unsigned* sig = (volatile LDSP unsigned*)(smem + NSLOT * SLOT_B + 16384 + 1280) + wv;
        *sig = 0;
#pragma unroll 1
        for (int ti = 0; ti < 2; ++ti) {
            const TileInfo d = decode(ti ? tile1 : tile0, rank);
            if (d.kind == T_NONE) break;
            unsigned cnt = __hip_atomic_load(mycnt, __ATOMIC_RELAXED, __HIP_MEMORY_SCOPE_SYSTEM);
            float sx = __hip_atomic_load(p.sx, __ATOMIC_RELAXED, __HIP_MEMORY_SCOPE_WORKGROUP);
            float sw = __hip_atomic_load(p.sw, __ATOMIC_RELAXED, __HIP_MEMORY_SCOPE_WORKGROUP);
            float bad = 0.f;
            if (__builtin_expect(d.kind == T_PEER, 0)) {
                if (!call) call = next_call(mycnt);
                if (tid < NXPART) {
                    wait_flag((unsigned*)(myws + WS_XFLAG) + d.peer * 32 + tid, call, tmo);
                    __builtin_amdgcn_fence(__ATOMIC_ACQUIRE, "");
                    asm volatile("s_waitcnt vmcnt(0)" ::: "memory");
                }
                asm volatile("s_barrier" ::: "memory");
                if (__hip_atomic_load(tmo, __ATOMIC_RELAXED, __HIP_MEMORY_SCOPE_AGENT) == call) bad = __builtin_nanf("");
            }
            const int srcr = d.kind == T_PEER ? d.peer : rank;
            char* ob = p.out[d.kind == T_REMOTE ? d.bb : NR - 1] + ((size_t)(srcr * MS + d.row0) * NS_COLS + d.y * BN) * 4;
            char* const pws = p.ws[d.bb];
            const bool fresh = !call;
            float scale, sfw;
            int eb, npk = d.npk, mt = d.mt;
            unsigned tsend = (unsigned)tstart + (pre >> 18);
            char* us;
            u64* fl;
            f32x4 v[8];
            auto hook = [&](float m) {
                asm volatile("" : "+v"(sx), "+v"(sw), "+v"(cnt));
                scale = sx * sw + bad;
                if (fresh) call = __builtin_amdgcn_readfirstlane(cnt) + 1;
                us = pws + WS_USTAGE + (call & 1) * USTAGE_PAR + rank * USTAGE_SRC + d.flag * UTILE + wv * mt * 3200 + lane * 16;
                fl = (u64*)(pws + WS_UFLAG + (call & 1) * UFLAG_PAR + rank * UFLAG_SRC) + (d.flag * 4 + wv) * 2;
                eb = (int)((__builtin_bit_cast(unsigned, m) >> 23) & 255) - 23;
                eb = __builtin_amdgcn_readfirstlane(eb < 8 ? 8 : eb);
                sfw = __builtin_bit_cast(float, (254 - eb) << 23);
                asm volatile("" : "+v"(us), "+v"(scale));
                asm volatile("" : "+s"(eb), "+s"(sfw), "+s"(fl), "+s"(ob), "+s"(npk), "+s"(tsend));
            };
            if (mt == 1) compute<1>(smem, wv, lane, v, hook);
            else         compute<2>(smem, wv, lane, v, hook);
#define ACK() do { } while (0)
            unsigned hdr = 0;
            PROF(0);
            u32x4 dat[2][4];
            bool rawu[2], sent[2];
            unsigned off = 0, step = 0;
            auto send_unit = [&](int h) {
                if (!rawu[h]) {
#pragma unroll
                    for (int u = 0; u < 3; ++u) { ACK(); st_wt(us + h * 3072 + u * 1024, dat[h][u]); }
                } else {
#pragma unroll
                    for (int u = 0; u < 4; ++u) { ACK(); st_wt(ob + off + (size_t)(4 * h + u) * step, dat[h][u]); }
                }
            };
#pragma unroll
            for (int h = 0; h < 2; ++h) {
                rawu[h] = true; sent[h] = true;
                if (h >= mt) break;
                bool raw = !npk;
                int q[16];
                u64 any = 0;
                if (!raw) {
                    float t[16], r[16];
#pragma unroll
                    for (int f = 0; f < 4; ++f) {
                        const f32x4 tf = v[4 * h + f] * sfw;
#pragma unroll
                        for (int e = 0; e < 4; ++e) t[4 * f + e] = tf[e];
                    }
#pragma unroll
                    for (int e = 0; e < 16; ++e) asm("v_cvt_i32_f32 %0, %1" : "=v"(q[e]) : "v"(t[e]));
#pragma unroll
                    for (int e = 0; e < 16; ++e) r[e] = (float)q[e];
                    unsigned miss[4] = {0, 0, 0, 0};
                    float big[2] = {0.f, 0.f};
#pragma unroll
                    for (int e = 0; e < 16; ++e)
                        miss[e & 3] = __builtin_amdgcn_bitop3_b32(miss[e & 3], __builtin_bit_cast(unsigned, r[e]), __builtin_bit_cast(unsigned, t[e]), 0xf6);
#pragma unroll
                    for (int e = 0; e < 8; ++e)
                        asm("v_max3_f32 %0, |%1|, |%2|, %3" : "=v"(big[e & 1]) : "v"(r[2 * e]), "v"(r[2 * e + 1]), "v"(big[e & 1]));
                    any = __builtin_amdgcn_ballot_w64((miss[0] | miss[1] | miss[2] | miss[3]) != 0) |
                          __builtin_amdgcn_ballot_w64(big[0] >= 16777216.f) | __builtin_amdgcn_ballot_w64(big[1] >= 16777216.f);
                    raw = eb > 126 || (int)__builtin_popcountll(any) > MAXESC;
                    if (!raw) {
                        if (any) {
#pragma unroll
                            for (int e = 0; e < 16; ++e) {
                                const float rc = __builtin_amdgcn_fmed3f(r[e], -16777215.f, 16777215.f);
                                q[e] = __builtin_bit_cast(unsigned, rc) != __builtin_bit_cast(unsigned, t[e]) ? -16777216 : q[e];
                            }
                        }
                        unsigned dw[12];
#pragma unroll
                        for (int f = 0; f < 4; ++f) {
                            dw[3 * f]     = __builtin_amdgcn_perm(q[4 * f + 1], q[4 * f], 0x04020100);
                            dw[3 * f + 1] = __builtin_amdgcn_perm(q[4 * f + 2], q[4 * f + 1], 0x05040201);
                            dw[3 * f + 2] = __builtin_amdgcn_perm(q[4 * f + 3], q[4 * f + 2], 0x06050402);
                        }
#pragma unroll
                        for (int u = 0; u < 3; ++u) dat[h][u] = (u32x4){dw[4 * u], dw[4 * u + 1], dw[4 * u + 2], dw[4 * u + 3]};
                    }
                }
                int ln = lane;
                asm volatile("" : "+v"(ln));
                if (raw) {
                    piece_geom(mt, wv, ln, off, step);
#pragma unroll
                    for (int f = 0; f < 4; ++f) *(LDSP f32x4*)(ep + frag_slot(mt, ln, f)) = v[4 * h + f] * scale;
#pragma unroll
                    for (int u = 0; u < 4; ++u) dat[h][u] = *(LDSP u32x4*)(ep + piece_slot(mt, ln, u));
                }
                rawu[h] = raw;
                hdr |= (raw ? 255u : (unsigned)eb | (any ? 128u : 0u)) << (8 * h);
                if (!h) PROF(1);
                sent[h] = (unsigned)__builtin_amdgcn_s_memrealtime() - tsend < 0x80000000u;
                if (sent[h]) send_unit(h);
                if (!h) PROF(2);
                if (!raw) {
                    unsigned pa = 0, pb = 0;
#pragma unroll
                    for (int e = 0; e < 8; ++e) { pa = __builtin_amdgcn_alignbit(pa, q[e], 31); pb = __builtin_amdgcn_alignbit(pb, q[8 + e], 31); }
                    *(LDSP unsigned short*)(pp + h * 128 + ln * 2) = (unsigned short)(pa << 8 | pb);
                    if (any) {
                        const int g = ln >> 4, li = ln & 15;
                        char* of = ob + (mt == 2 ? (32 * (wv >> 1) + 16 * h + li) * NS_COLS * 4 + (wv & 1) * 256 : li * NS_COLS * 4 + wv * 128) + g * 16;
                        const unsigned f1 = 64, f2 = mt == 2 ? 128 : 16 * NS_COLS * 4;
#pragma unroll
                        for (int f = 0; f < 4; ++f) {
                            if (__builtin_amdgcn_ballot_w64(q[4 * f] == -16777216 || q[4 * f + 1] == -16777216 || q[4 * f + 2] == -16777216 || q[4 * f + 3] == -16777216)) {
#pragma unroll
                                for (int e = 0; e < 4; ++e) {
                                    const float x = v[4 * h + f][e];
                                    if (q[4 * f + e] == -16777216) st_wt1(of + (f & 1) * f1 + (f >> 1) * f2 + 4 * e, x * scale);
                                }
                            }
                        }
                    }
                }
                if (!h) PROF(3);
            }
            u32x4 pv;
            if (npk) pv = *(LDSP u32x4*)(pp + (lane & 15) * 16);
            PROF(4);
            while ((unsigned)__builtin_amdgcn_s_memrealtime() - tsend > 0x80000000u) __builtin_amdgcn_s_sleep(1);
            PROF(5);
            if (!sent[0]) send_unit(0);
            if (!sent[1]) send_unit(1);
            if (npk && lane < 8 * mt) st_wt(us + mt * 3072, pv);
            if (ti == 0 && (tile1 & 3) != T_NONE) asm volatile("s_barrier" ::: "memory");
            if (d.kind == T_REMOTE) {
                *sig = 1;
                PROF(6);
                asm volatile("s_waitcnt vmcnt(0)" ::: "memory");
                if (lane == 0) __hip_atomic_store(fl, (u64)call << 32 | hdr, __ATOMIC_RELAXED, __HIP_MEMORY_SCOPE_SYSTEM);
                PROF(7);
            }
            if (fresh && lane == 0) __hip_atomic_store(mycnt, call, __ATOMIC_RELAXED, __HIP_MEMORY_SCOPE_SYSTEM);
            if (ti == 0) STAMP(3); else STAMP(7);
            if (ti == 0) PROFOUT(0, 8);
        }
    }

    {
        unsigned tw = tile0;
        asm volatile("" : "+s"(tw));
        const TileInfo d = decode(tw, rank);
        const int h = wv < 4, wq = wv & 3;
        if (d.kind == T_REMOTE && (!h || (d.npk && d.mt == 2))) {
            int ln = lane;
            asm volatile("" : "+v"(ln));
            const int src = (rank - 1 - d.bb) & (NR - 1);
            float sc = *p.sx * *p.sw;
            unsigned off, step;
            piece_geom(d.mt, wq, ln, off, step);
            char* o = p.out[NR - 1] + ((size_t)(src * MS + d.row0) * NS_COLS + d.y * BN) * 4 + off + 4 * h * step;
            char* o2 = o + 2 * step;
            const char* us = myws + WS_USTAGE + (call & 1) * USTAGE_PAR + src * USTAGE_SRC + d.flag * UTILE + wq * d.mt * 3200;
            u64* fl = (u64*)(myws + WS_UFLAG + (call & 1) * UFLAG_PAR + src * UFLAG_SRC) + (d.flag * 4 + wq) * 2;
            const char* a0 = us + h * 3072 + ln * 16;
            const char* a1 = us + d.mt * 3072 + h * 128 + ln * 2;
            const unsigned rp = 2 * SLOT_B + wv * 4096;
            unsigned fs[4], ps[2];
#pragma unroll
            for (int f = 0; f < 4; ++f) fs[f] = rp + frag_slot(d.mt, ln, f);
            ps[0] = rp + piece_slot(d.mt, ln, 0); ps[1] = rp + piece_slot(d.mt, ln, 1);
            char* o1 = o + step;
            char* o3 = o2 + step;
            asm volatile("" : "+v"(a0), "+v"(a1), "+v"(o), "+v"(o1), "+v"(o2), "+v"(o3), "+v"(sc),
                              "+v"(fs[0]), "+v"(fs[1]), "+v"(fs[2]), "+v"(fs[3]), "+v"(ps[0]), "+v"(ps[1]));
            unsigned hdr = 0;
            volatile LDSP unsigned* sig = (volatile LDSP unsigned*)(smem + NSLOT * SLOT_B + 16384 + 1280) + wq;
            while (__builtin_amdgcn_readfirstlane(*sig) == 0) __builtin_amdgcn_s_sleep(4);
            PROF(0);
            const bool ok = wait_uflag(fl, call, tmo, hdr);
            PROF(1);
            const unsigned hb = (hdr >> (8 * h)) & 255;
            if (!ok) st_wt1(o, __builtin_nanf(""));
            else if (hb != 255) {
                u32x4 a, b, c;
                unsigned pl;
                asm volatile("global_load_dwordx4 %0, %4, off sc0 sc1\n\t"
                             "global_load_dwordx4 %1, %4, off offset:1024 sc0 sc1\n\t"
                             "global_load_dwordx4 %2, %4, off offset:2048 sc0 sc1\n\t"
                             "global_load_ushort %3, %5, off sc0 sc1\n\ts_waitcnt vmcnt(0)"
                             : "=&v"(a), "=&v"(b), "=&v"(c), "=&v"(pl) : "v"(a0), "v"(a1) : "memory");
                PROF(2);
                const unsigned dw[12] = {a.x, a.y, a.z, a.w, b.x, b.y, b.z, b.w, c.x, c.y, c.z, c.w};
                const float sinv = __builtin_bit_cast(float, ((hb & 127) - 7) << 23);
                u32x4 t[4], sg[4];
#pragma unroll
                for (int f = 0; f < 4; ++f) {
                    t[f] = (u32x4){dw[3 * f] << 8, __builtin_amdgcn_perm(dw[3 * f + 1], dw[3 * f], 0x0504030c),
                                   __builtin_amdgcn_perm(dw[3 * f + 2], dw[3 * f + 1], 0x0403020c), dw[3 * f + 2] & 0xffffff00u};
                    sg[f] = (u32x4){pl >> (15 - 4 * f), pl >> (14 - 4 * f), pl >> (13 - 4 * f), pl >> (12 - 4 * f)};
                }
                asm volatile("" : "+v"(t[0]), "+v"(t[1]), "+v"(t[2]), "+v"(t[3]), "+v"(sg[0]), "+v"(sg[1]), "+v"(sg[2]), "+v"(sg[3]));
#pragma unroll
                for (int f = 0; f < 4; ++f) {
                    t[f] = (u32x4){__builtin_amdgcn_alignbit(sg[f].x, t[f].x, 1), __builtin_amdgcn_alignbit(sg[f].y, t[f].y, 1),
                                   __builtin_amdgcn_alignbit(sg[f].z, t[f].z, 1), __builtin_amdgcn_alignbit(sg[f].w, t[f].w, 1)};
                    *(LDSP u32x4*)(smem + fs[f]) = t[f];
                }
#pragma unroll
                for (int u = 0; u < 4; ++u) t[u] = *(LDSP u32x4*)(smem + ps[u & 1] + (u >> 1) * 2048);
                char* const op[4] = {o, o1, o2, o3};
                if (!(hb & 128)) {
#pragma unroll
                    for (int u = 0; u < 4; ++u) {
                        const f32x4 r = ((f32x4){(float)(int)t[u].x, (float)(int)t[u].y, (float)(int)t[u].z, (float)(int)t[u].w} * sinv) * sc;
                        st_wt(op[u], __builtin_bit_cast(u32x4, r));
                    }
                } else {
#pragma unroll
                    for (int u = 0; u < 4; ++u) {
                        const f32x4 r = ((f32x4){(float)(int)t[u].x, (float)(int)t[u].y, (float)(int)t[u].z, (float)(int)t[u].w} * sinv) * sc;
                        const int tj[4] = {(int)t[u].x, (int)t[u].y, (int)t[u].z, (int)t[u].w};
                        const bool hole = min(min(tj[0], tj[1]), min(tj[2], tj[3])) == (int)0x80000000u;
                        if (!hole) st_wt(op[u], __builtin_bit_cast(u32x4, r));
                        if (__builtin_amdgcn_ballot_w64(hole)) {
#pragma unroll
                            for (int j = 0; j < 4; ++j) { const float rj = r[j]; if (hole && tj[j] != (int)0x80000000u) st_wt1(op[u] + 4 * j, rj); }
                        }
                    }
                }
            }
            PROF(4);
        }
    }
}

#define PREF 1
#define PACE 0
#define DELAYL 400
#define PK 1
#define XON7 0
#define KSTAG 10
#define NRAW 3
#define TFIRST 700
#define RATE 57.0
#define TLEAD 950
static unsigned pack_pre(int kind, int ticks, int job, int send) { return kind | ticks << 2 | job << 13 | (unsigned)send << 18; }
static void make_plan(Params& p) {
    for (int b = 0; b < GRID; ++b) {
        const int x = b & 7, i = b >> 3;
        Prog& g = p.prog[b];
        g.pre = PRE_NONE; g.t[0] = g.t[1] = 0;
        if (x < 7) {
            const double u32 = PK ? 12.9 : 16.3, u64p = PK ? 25.5 : 32.3, u64r = 32.3;
            if (i >= 4 && i < 12) {
                g.pre = pack_pre(PRE_NONE, 0, 0, TFIRST + (int)((i - 4) * u32 * 100 / RATE));
                g.t[0] = pack_tile(p.rank, T_REMOTE, 1, 2 * ((i - 4) >> 1), x, i & 1, 0, i, PK);
                g.t[1] = pack_tile(p.rank, T_OWN, 1, 16 * (x & 1) + 2 * (i - 4), 7, x >> 1, 0, 0, 0);
            } else {
                const int k = i < 4 ? 20 + i : i - 12;
                const int nr = k > 24 - NRAW ? k - (24 - NRAW) : 0;
                const int send = TFIRST + (int)((8 * u32 + (k - nr) * u64p + nr * u64r) * 100 / RATE);
                const int start = k >= KSTAG && send > TLEAD + 50 ? send - TLEAD : 0;
                const int job = PREF && k >= 20 - NPF ? k - (20 - NPF) + 1 : 0;
                if (i < 4)      g.pre = pack_pre(PRE_XSEND, start, i | x << 2, send);
                else if (start) g.pre = pack_pre(PRE_DELAY, start, job, send);
                else            g.pre = pack_pre(PRE_NONE, 0, 0, send);
                const int npk = k >= 24 - NRAW ? 0 : PK;
                if (k < 10) g.t[0] = pack_tile(p.rank, T_REMOTE, 2, 8 + 4 * (k >> 1), x, k & 1, 0, i, npk);
                else        g.t[0] = pack_tile(p.rank, T_REMOTE, 2, 4 * ((k - 10) % 7), x, 2 + (k - 10) / 7, 0, i, npk);
            }
        } else {
            g.pre = pack_pre(PRE_DELAY, DELAYL, 16, 0);
            if (i < 28) g.t[0] = pack_tile(p.rank, T_PEER, 2, 28, i >> 2, i & 3, 8 * (i & 3), 0, 0);
            else        g.t[0] = pack_tile(p.rank, T_OWN, 2, 16 + 4 * (i - 28), 7, 3, 0, 0, 0);
        }
    }
}

extern "C" void kernel_launch(int rank, int nranks, ncclComm_t comm,
                              hipStream_t stream,
                              void* const* d_in, const int* in_sizes, int n_in,
                              void* d_out, int out_size,
                              const int* in_dtypes, int out_dtype,
                              void* const* d_out_all,
                              void* const* d_ws_all, size_t ws_bytes) {
    if (nranks != NR || (unsigned)rank >= (unsigned)NR || n_in < 4 || ws_bytes < WS_NEED ||
        in_sizes[0] != MS * KD || in_sizes[1] != KD * ND || out_size != NR * MS * NS_COLS) {
        fprintf(stderr, "kernel_launch: unexpected problem shape\n");
        return;
    }
    Params p;
    p.x = (const char*)d_in[0];
    p.w = (const char*)d_in[1];
    p.sx = (const float*)d_in[2];
    p.sw = (const float*)d_in[3];
    for (int b = 0; b < NR; ++b) { p.out[b] = (char*)d_out_all[(rank + 1 + b) % NR]; p.ws[b] = (char*)d_ws_all[(rank + 1 + b) % NR]; }
    p.rank = rank;
    p.pad = 0;
    make_plan(p);
    gemm_a2a<<<GRID, NTHR, 0, stream>>>(p);
}
```

```cpp
#include <hip/hip_runtime.h>
#include <rccl/rccl.h>
#include <cstddef>
#include <cstdio>

typedef unsigned u32x4 __attribute__((ext_vector_type(4)));
typedef int i32x2 __attribute__((ext_vector_type(2)));
typedef int i32x4 __attribute__((ext_vector_type(4)));
typedef int i32x8 __attribute__((ext_vector_type(8)));
typedef float f32x4 __attribute__((ext_vector_type(4)));
typedef unsigned long long u64;
#define LDSP __attribute__((address_space(3)))
#define GLBP __attribute__((address_space(1)))

constexpr int NR = 8;
constexpr int KD = 4096, ND = 4096;
constexpr int MS = 512, NS_COLS = 512;
constexpr int BN = 128, BK = 128;
constexpr int NKS = KD / BK;
#define NSLOT_ 4
#define PF7T 30
#define PFWAIT 0
#define PK 1
#define NRAW 0
#define RATE 58.0
#define TFIRST 705
#define SLIPADD 30
#define NEAR 75
constexpr int CODE_BYTES = 21504;
#define XLOC 3
#define XPOS 1
#define PSLEEP 10
#define NPULL 6
#define PLEAD 75
#define DUE 35
constexpr int NSLOT = NSLOT_;
constexpr int NTHR = 512;
constexpr int SLOT_B = (64 + BN) * BK;
#define XROWS_ 96
constexpr int XROWS = XROWS_;
constexpr int NXS = XROWS / 8;
constexpr int NXPART = 4;
constexpr int GRID = 256;
constexpr int NTILE = 32;
constexpr u64 DEADLINE_TICKS = 50000000ull;

constexpr size_t WS_CNT = 0;
constexpr size_t WS_TMO = 8192;
constexpr size_t WS_LINK = 9216;
constexpr size_t WS_UFLAG = 524288;
constexpr size_t UFLAG_SRC = NTILE * 4 * 64, UFLAG_PAR = NR * UFLAG_SRC;
constexpr int NESC = 7;
constexpr size_t WS_XSTAGE = 1u << 20;
constexpr size_t XSTAGE_SRC = (size_t)XROWS * KD, XSTAGE_PAR = NR * XSTAGE_SRC;
constexpr size_t WS_XFLAG = WS_XSTAGE + 2 * XSTAGE_PAR;
constexpr size_t WS_USTAGE = 12u << 20;
constexpr int UTILE = 4 * 2 * 3200;
constexpr size_t USTAGE_SRC = 1u << 20, USTAGE_PAR = NR * USTAGE_SRC;
static_assert(NTILE * UTILE <= USTAGE_SRC, "staging");
constexpr size_t PULL_OFF = 2 * USTAGE_PAR;
constexpr size_t WS_NEED = WS_USTAGE + 4 * USTAGE_PAR;
constexpr int RP_SLOT = (NKS - 1) % NSLOT >= 2 ? 0 : 2;
static_assert(NSLOT >= 4, "the receivers' patches take two ring slots");
static_assert(WS_UFLAG + 2 * UFLAG_PAR <= WS_XSTAGE && WS_XSTAGE + 3 * XSTAGE_PAR <= WS_USTAGE, "workspace layout");

enum { PRE_NONE = 0, PRE_XSEND = 1, PRE_DELAY = 2 };
enum { T_NONE = 0, T_REMOTE = 1, T_OWN = 2, T_PEER = 3 };
struct Prog {
    unsigned pre;
    unsigned t[2];
};
static unsigned pack_tile(int rank, int kind, int mt, int row, int bb, int y, int rot, int nr, int flag, int pk, int own) {
    const unsigned cb = (kind == T_REMOTE ? (rank + 1 + bb) % NR : rank) * 4 + y;
    return kind | (mt - 1) << 2 | row << 3 | bb << 8 | y << 11 | rot << 13 | (nr - 1) << 18 | flag << 20 | pk << 25 | own << 26 | cb << 27;
}
struct TileInfo { int kind, mt, row, y, bb, rot, pull, nr, flag, pk, own; };
__device__ __forceinline__ TileInfo decode(unsigned t) {
    TileInfo d;
    d.kind = t & 3; d.mt = ((t >> 2) & 1) + 1; d.row = (t >> 3) & 31; d.bb = (t >> 8) & 7; d.y = (t >> 11) & 3;
    d.rot = d.kind == T_REMOTE ? 0 : (t >> 13) & 31; d.pull = d.kind == T_REMOTE ? (t >> 13) & 1 : 0; d.nr = ((t >> 18) & 3) + 1; d.flag = (t >> 20) & 31; d.pk = (t >> 25) & 1; d.own = (t >> 26) & 1;
    return d;
}

struct Link {
    char* xdst;
    char* ustage;
    char* uflag;
    char* out;
};
struct Params {
    const char* x;
    const char* w;
    char* ws;
    int rank;
    int pad;
    const float* sx;
    const float* sw;
    char* out;
    char* pad2;
    Link link[8];
    Prog prog[GRID];
};

__device__ __forceinline__ void touch(unsigned lds, unsigned voff, const char* sbase) {
    asm volatile("s_mov_b32 m0, %2\n\ts_nop 0\n\tglobal_load_lds_dword %0, %1" :: "v"(voff), "s"(sbase), "s"(lds) : "memory");
}
__device__ __forceinline__ void st_wt(char* p, u32x4 v) {
    asm volatile("global_store_dwordx4 %0, %1, off sc0 sc1\n\ts_nop 1" :: "v"(p), "v"(v) : "memory");
}
__device__ __forceinline__ void st_wt1(char* p, float v) {
    asm volatile("global_store_dword %0, %1, off sc0 sc1" :: "v"(p), "v"(v) : "memory");
}
__device__ __forceinline__ void wait_flag(unsigned* flag, unsigned call, unsigned* tmo) {
    u64 t0 = __builtin_amdgcn_s_memrealtime();
    while (__hip_atomic_load(flag, __ATOMIC_RELAXED, __HIP_MEMORY_SCOPE_SYSTEM) < call) {
        if (__hip_atomic_load(tmo, __ATOMIC_RELAXED, __HIP_MEMORY_SCOPE_AGENT) == call ||
            __builtin_amdgcn_s_memrealtime() - t0 > DEADLINE_TICKS) {
            __hip_atomic_store(tmo, call, __ATOMIC_RELAXED, __HIP_MEMORY_SCOPE_AGENT);
            break;
        }
        __builtin_amdgcn_s_sleep(2);
    }
}
__device__ __forceinline__ bool wait_uflag(const u64* word, unsigned key, unsigned lim, unsigned last, unsigned call, unsigned* tmo, u64& f) {
    const u64 t0 = __builtin_amdgcn_s_memrealtime();
    for (;;) {
        unsigned left, t;
        asm volatile("s_mov_b32 %1, 64\n"
                     "1:\n\t"
                     "global_load_dwordx2 v[40:41], %3, off sc0 sc1\n\t"
                     "s_waitcnt vmcnt(0)\n\t"
                     "v_xor_b32 %2, %4, v41\n\t"
                     "v_cmp_lt_u32 vcc, %5, %2\n\t"
                     "s_cbranch_vccz 2f\n\t"
                     "s_sub_u32 %1, %1, 1\n\t"
                     "s_cmp_lg_u32 %1, 0\n\t"
                     "s_cbranch_scc1 1b\n\t"
                     "s_branch 3f\n"
                     "2:\n\t"
                     "s_cmp_eq_u32 %6, 0\n\t"
                     "s_cbranch_scc1 3f\n\t"
                     "s_endpgm\n"
                     "3:"
                     : "=&{v[40:41]}"(f), "=&s"(left), "=&v"(t) : "v"(word), "v"(key), "v"(lim), "s"(last) : "memory", "vcc", "scc");
        if (__builtin_expect(__builtin_amdgcn_readfirstlane(left) != 0, 1)) return true;
        if (__hip_atomic_load(tmo, __ATOMIC_RELAXED, __HIP_MEMORY_SCOPE_AGENT) == call ||
            __builtin_amdgcn_s_memrealtime() - t0 > DEADLINE_TICKS) {
            __hip_atomic_store(tmo, call, __ATOMIC_RELAXED, __HIP_MEMORY_SCOPE_AGENT);
            return false;
        }
        __builtin_amdgcn_s_sleep(8);
    }
}
#define STAMP(i) do { } while (0)
#define LSTAMP(i, v) do { } while (0)
#define PROF(i) do { } while (0)
#define PROFOUT(base, n) do { } while (0)

template <int MT> struct Frags { i32x8 x[2], w[2 * MT]; };

template <int MT>
__device__ __forceinline__ void read_frags(Frags<MT>& f, LDSP char* sl, const unsigned (&xrd)[2], const unsigned (&wrd)[2 * MT]) {
#pragma unroll
    for (int b = 0; b < 2 * MT; ++b) {
        i32x2 t0 = __builtin_amdgcn_ds_read_tr8_b64_v2i32((LDSP i32x2*)(sl + wrd[b]));
        i32x2 t1 = __builtin_amdgcn_ds_read_tr8_b64_v2i32((LDSP i32x2*)(sl + wrd[b] + 8 * BN));
        i32x2 t2 = __builtin_amdgcn_ds_read_tr8_b64_v2i32((LDSP i32x2*)(sl + wrd[b] + 64 * BN));
        i32x2 t3 = __builtin_amdgcn_ds_read_tr8_b64_v2i32((LDSP i32x2*)(sl + wrd[b] + 72 * BN));
        f.w[b] = (i32x8){t0.x, t0.y, t1.x, t1.y, t2.x, t2.y, t3.x, t3.y};
        if (b == 0) {
#pragma unroll
            for (int a = 0; a < 2; ++a) {
                i32x4 lo = *(LDSP i32x4*)(sl + xrd[0] + a * 4096);
                i32x4 hi = *(LDSP i32x4*)(sl + xrd[1] + a * 4096);
                f.x[a] = __builtin_shufflevector(lo, hi, 0, 1, 2, 3, 4, 5, 6, 7);
            }
        }
    }
}
template <int MT>
__device__ __forceinline__ void mma(f32x4 (&acc)[2][2 * MT], const Frags<MT>& f) {
#pragma unroll
    for (int b = 0; b < 2 * MT; ++b)
#pragma unroll
        for (int a = 0; a < 2; ++a)
            acc[a][b] = __builtin_amdgcn_mfma_scale_f32_16x16x128_f8f6f4(f.w[b], f.x[a], acc[a][b], 1, 1, 0, 0, 0, 0);
}
template <int MT>
__device__ __forceinline__ void interleave() {
#pragma unroll
    for (int u = 0; u < 2 * MT; ++u) {
        __builtin_amdgcn_sched_group_barrier(0x008, 1, 0);
        __builtin_amdgcn_sched_group_barrier(0x100, MT == 2 ? 5 : 6, 0);
    }
    __builtin_amdgcn_sched_group_barrier(0x008, 2 * MT, 0);
}
#define CBAR() do { __builtin_amdgcn_sched_barrier(0); __builtin_amdgcn_s_waitcnt(0xC07F); \
                    asm volatile("s_waitcnt lgkmcnt(0)\n\ts_barrier" ::: "memory"); __builtin_amdgcn_sched_barrier(0); } while (0)
constexpr int AHEAD = NSLOT - 1;

#define LD_ISSUE(XTRA) \
    "s_mov_b32 m0, %[slot]\n\t" \
    "s_lshr_b32 %[sox], %[sow], 12\n\t" \
    "buffer_load_dwordx4 %[w0], %[rw], %[sow] offen lds\n\t" \
    "buffer_load_dwordx4 %[w1], %[rw], %[sow] offen offset:1024 lds\n\t" \
    "buffer_load_dwordx4 %[w2], %[rw], %[sow] offen offset:2048 lds\n\t" \
    "buffer_load_dwordx4 %[w3], %[rw], %[sow] offen offset:3072 lds\n\t" \
    "s_add_u32 m0, %[slot], %[ldsx]\n\t" \
    "s_add_u32 %[slot], %[slot], %[slotb]\n\t" \
    "buffer_load_dwordx4 %[x0], %[rx], %[sox] offen sc0 sc1 lds\n\t" \
    XTRA \
    "s_cmp_eq_u32 %[slot], %[send]\n\t" \
    "s_cselect_b32 %[slot], %[sbeg], %[slot]\n\t" \
    "s_add_u32 %[sow], %[sow], 0x80000\n\t" \
    "s_and_b32 %[sow], %[sow], 0xffffff\n\t"
struct Stream { unsigned w0, w1, w2, w3, x0, x1, ldsw, ldsx; };
#define LD_OPS \
    [w0] "v"(k.w0), [w1] "v"(k.w1), [w2] "v"(k.w2), [w3] "v"(k.w3), [x0] "v"(k.x0), [x1] "v"(k.x1), \
    [rw] "s"(rw), [rx] "s"(rx), [ldsx] "s"(k.ldsx), [sbeg] "s"(k.ldsw), [send] "s"(k.ldsw + NSLOT * SLOT_B), [slotb] "i"(SLOT_B)
#define LD_X1 "buffer_load_dwordx4 %[x1], %[rx], %[sox] offen offset:1024 sc0 sc1 lds\n\t"
__device__ __forceinline__ u32x4 rsrc(const char* base, unsigned bytes) {
    return (u32x4){(unsigned)(u64)base, (unsigned)((u64)base >> 32), bytes, 0x00020000u};
}
__device__ __forceinline__ void stream_start(const Stream& k, const char* wbase, const char* xbase, unsigned xbytes, unsigned rot, unsigned big) {
    const u32x4 rw = rsrc(wbase, KD * ND), rx = rsrc(xbase, xbytes);
    unsigned slot = k.ldsw, sow = rot * (BK * ND), sox, n;
    asm volatile("s_nop 1\n\ts_mov_b32 %[n], %[ahead]\n"
                 "1:\n\t" LD_ISSUE("s_cmp_eq_u32 %[big], 0\n\ts_cbranch_scc1 3f\n\t" LD_X1 "3:\n\t")
                 "s_sub_u32 %[n], %[n], 1\n\t"
                 "s_cmp_lg_u32 %[n], 0\n\t"
                 "s_cbranch_scc1 1b"
                 : [slot] "+&s"(slot), [sow] "+&s"(sow), [sox] "=&s"(sox), [n] "=&s"(n)
                 : LD_OPS, [ahead] "i"(AHEAD), [big] "s"(big) : "memory", "scc");
}
#define LD_MAIN(XTRA) \
    "s_mov_b32 %[n], %[nmain]\n" \
    "2:\n\t" \
    "s_waitcnt vmcnt(%[c2])\n\t" \
    "s_barrier\n\t" LD_ISSUE(XTRA) \
    "s_sub_u32 %[n], %[n], 1\n\t" \
    "s_cmp_lg_u32 %[n], 0\n\t" \
    "s_cbranch_scc1 2b"
template <int MT>
__device__ __forceinline__ void stream_rest(const Stream& k, const char* wbase, const char* xbase, unsigned xbytes, unsigned rot) {
    const u32x4 rw = rsrc(wbase, KD * ND), rx = rsrc(xbase, xbytes);
    unsigned slot = k.ldsw + AHEAD * SLOT_B, sow = ((rot + AHEAD) & (NKS - 1)) * (BK * ND), sox, n;
    constexpr int PER = 4 + MT;
    if (MT == 2)
        asm volatile(LD_MAIN(LD_X1) : [slot] "+&s"(slot), [sow] "+&s"(sow), [sox] "=&s"(sox), [n] "=&s"(n)
                     : LD_OPS, [nmain] "i"(NKS - AHEAD), [c2] "i"(PER * (AHEAD - 1)) : "memory", "scc");
    else
        asm volatile(LD_MAIN("") : [slot] "+&s"(slot), [sow] "+&s"(sow), [sox] "=&s"(sox), [n] "=&s"(n)
                     : LD_OPS, [nmain] "i"(NKS - AHEAD), [c2] "i"(PER * (AHEAD - 1)) : "memory", "scc");
#pragma unroll
    for (int s = AHEAD - 1; s >= 0; --s)
        asm volatile("s_waitcnt vmcnt(%0)\n\ts_barrier" :: "i"(PER * s) : "memory");
}

template <int CTRL, int ROWS>
__device__ __forceinline__ float dpp_add(float v) {
    return v + __builtin_bit_cast(float, __builtin_amdgcn_update_dpp(0, __builtin_bit_cast(int, v), CTRL, ROWS, 0xf, true));
}
template <int MT>
__device__ __forceinline__ void compute(LDSP char* smem, int wv, int lane, f32x4 (&out)[8], int& eb, unsigned& tlast, unsigned& kp) {
    const int g = lane >> 4, li = lane & 15;
    const int wm = MT == 2 ? wv >> 1 : 0, wc = MT == 2 ? 4 * (wv & 1) : 2 * wv;
    unsigned xrd[2], wrd[2 * MT];
    const int fx = (li >> 1) & 7;
    xrd[0] = BN * BK + (li >> 3) * 2048 + wm * 1024 + (li & 7) * BK + ((g ^ fx) << 4);
    xrd[1] = xrd[0] ^ 64;
    const int q = li >> 1, pp = li & 1;
    const int fw = ((q >> 1) & 3) | ((g & 1) << 2);
#pragma unroll
    for (int b = 0; b < 2 * MT; ++b) wrd[b] = (16 * g + q) * BN + (((wc + b) ^ fw) << 4) + 8 * pp;
    f32x4 acc[2][2 * MT];
#pragma unroll
    for (int a = 0; a < 2; ++a)
#pragma unroll
        for (int b = 0; b < 2 * MT; ++b) acc[a][b] = (f32x4){0.f, 0.f, 0.f, 0.f};
    Frags<MT> fa, fb;
    float msum = 0.f;
    unsigned slot = 0;
    auto next = [&]() { LDSP char* r = smem + slot; slot = slot + SLOT_B == NSLOT * SLOT_B ? 0 : slot + SLOT_B; return r; };
    CBAR();
    read_frags<MT>(fa, next(), xrd, wrd);
    auto two_stages = [&](int s) {
        CBAR();
        read_frags<MT>(fb, next(), xrd, wrd);
        mma<MT>(acc, fa);
        interleave<MT>();
        CBAR();
        read_frags<MT>(fa, next(), xrd, wrd);
        mma<MT>(acc, fb);
        interleave<MT>();
        __builtin_amdgcn_sched_barrier(0);
    };
#pragma unroll 1
    for (int s = 1; s < NKS - 9; s += 2) two_stages(s);
    asm volatile("" : "+s"(slot));
#pragma unroll 1
    for (int s = NKS - 9; s < NKS - 1; s += 2) {
        two_stages(s);
        if (s == NKS - 9) {
            const float a0 = acc[0][0][0], a1 = acc[0][0][1], a2 = acc[0][0][2], a3 = acc[0][0][3];
            msum = __builtin_fabsf(a0) + __builtin_fabsf(a1) + __builtin_fabsf(a2) + __builtin_fabsf(a3);
            msum = dpp_add<0x118, 0xf>(dpp_add<0x114, 0xf>(dpp_add<0x112, 0xf>(dpp_add<0x111, 0xf>(msum))));
        }
        if (s == NKS - 7) {
            tlast = (unsigned)__builtin_amdgcn_s_memrealtime();
            msum = dpp_add<0x143, 0xc>(dpp_add<0x142, 0xa>(msum));
            const int e = (int)((((unsigned)__builtin_amdgcn_readlane(__builtin_bit_cast(int, msum), 63) + 0x180000u) >> 23) & 255) - 29;
            eb = e < 8 ? 8 : e;
        }
    }
    asm volatile("" : "+s"(slot));
    CBAR();
    read_frags<MT>(fb, next(), xrd, wrd);
    mma<MT>(acc, fa);
    interleave<MT>();
    __builtin_amdgcn_sched_barrier(0);
#pragma unroll
    for (int a = 0; a < 2; ++a)
#pragma unroll
        for (int b = 0; b < 2 * MT; ++b)
            out[2 * MT * a + b] = __builtin_amdgcn_mfma_scale_f32_16x16x128_f8f6f4(fb.w[b], fb.x[a], acc[a][b], 1, 1, 0, 0, 0, 0);
    if (MT == 1) asm volatile("" : "=v"(out[4]), "=v"(out[5]), "=v"(out[6]), "=v"(out[7]));
}
__device__ __forceinline__ unsigned frag_slot(int mt, int lane, int f) {
    const int g = lane >> 4, li = lane & 15;
    return mt == 2 ? li * 256 + (((4 * f + g) ^ (li & 7)) << 4) : (16 * (f >> 1) + li) * 128 + (((4 * (f & 1) + g) ^ (li & 7)) << 4);
}
__device__ __forceinline__ unsigned piece_slot(int mt, int lane, int u) {
    return u * 1024 + ((lane ^ (mt == 2 ? (4 * u + (lane >> 4)) & 7 : lane >> 3)) << 4);
}
__device__ __forceinline__ void piece_geom(int mt, int lane, unsigned& off, unsigned& step) {
    off = mt == 2 ? (lane >> 4) * NS_COLS * 4 + (lane & 15) * 16 : (lane >> 3) * NS_COLS * 4 + (lane & 7) * 16;
    step = (mt == 2 ? 4 : 8) * NS_COLS * 4;
}
__device__ __forceinline__ unsigned unit_col(int mt, int y, int wq) { return y * BN * 4 + (mt == 2 ? (wq & 1) * 256 : wq * 128); }
__device__ __forceinline__ int unit_band(int mt, int wq, int h) { return mt == 2 ? 2 * (wq >> 1) + h : 0; }

#define XHEAD() \
    asm volatile("s_and_b32 s61, %[bid], 0xe7\n\t"                \
                 "s_xor_b32 s62, s61, %[xloc]\n\t" \
                 "s_sub_u32 s62, s62, 1\n\t" \
                 "s_cmp_ge_u32 s62, 7\n\t" \
                 "s_cbranch_scc1 .Lxret\n\t" \
                 "s_lshl_b32 s62, %[bid], 10\n\t" \
                 "s_and_b32 s62, s62, 0x6000\n\t" \
                 "s_or_b32 s62, s62, %[xoff]\n\t" \
                 "v_lshl_or_b32 v10, %[tid], 4, s62\n\t" \
                 "s_lshl_b32 s63, %[bid], 5\n\t" \
                 "v_add_u32 v11, 0x8000, v10\n\t" \
                 "v_cmp_eq_u32 vcc, 0, %[tid]\n\t" \
                 "s_waitcnt lgkmcnt(0)\n\t" \
                 "global_load_dwordx4 v[2:5], v10, s[20:21]\n\t" \
                 "global_load_dwordx4 v[6:9], v11, s[20:21]\n\t" \
                 "s_load_dword s60, s[24:25], s63\n\t"            \
                 "s_branch .Lxbody\n" \
                 ".Lxret:" \
                 : "+{s[20:27]}"(hot), "+{s[64:71]}"(lnk) \
                 : [bid] "s"(blockIdx.x), [tid] "v"(threadIdx.x), [xloc] "i"(XLOC), [xoff] "i"((MS - XROWS) * KD) \
                 : "memory", "scc", "vcc", "s58", "s59", "s60", "s61", "s62", "s63", \
                   "v2", "v3", "v4", "v5", "v6", "v7", "v8", "v9", "v10", "v11")
#define XBODY() \
    asm volatile("s_branch .Lxover\n" \
                 ".Lxbody:\n\t" \
                 "s_mov_b32 s61, %[n]\n\t" \
                 "s_waitcnt lgkmcnt(0)\n\t" \
                 "s_add_u32 s60, s60, 1\n\t" \
                 "s_bitcmp1_b32 s60, 0\n\t" \
                 "s_cselect_b32 s62, %[par], 0\n\t" \
                 "s_add_u32 s58, s64, s62\n\t" \
                 "s_addc_u32 s59, s65, 0\n\t" \
                 "s_waitcnt vmcnt(1)\n" \
                 "1:\n\t" \
                 "global_store_dwordx4 v10, v[2:5], s[58:59] sc0 sc1\n\t" \
                 "v_add_u32 v10, 0x10000, v10\n\t" \
                 "s_sub_u32 s61, s61, 1\n\t" \
                 "global_load_dwordx4 v[2:5], v10, s[20:21]\n\t" \
                 "s_waitcnt vmcnt(2)\n\t" \
                 "global_store_dwordx4 v11, v[6:9], s[58:59] sc0 sc1\n\t" \
                 "v_add_u32 v11, 0x10000, v11\n\t" \
                 "s_cmp_lg_u32 s61, 0\n\t" \
                 "global_load_dwordx4 v[6:9], v11, s[20:21]\n\t" \
                 "s_waitcnt vmcnt(2)\n\t" \
                 "s_cbranch_scc1 1b\n\t" \
                 "global_store_dwordx4 v10, v[2:5], s[58:59] sc0 sc1\n\t" \
                 "s_waitcnt vmcnt(1)\n\t" \
                 "global_store_dwordx4 v11, v[6:9], s[58:59] sc0 sc1\n\t" \
                 "s_lshr_b32 s63, s63, 6\n\t"                     \
                 "s_and_b32 s63, s63, 12\n\t" \
                 "s_add_u32 s63, s63, %[flag]\n\t" \
                 "v_mov_b32 v10, s63\n\t" \
                 "v_mov_b32 v11, s60\n\t" \
                 "s_waitcnt vmcnt(0)\n\t" \
                 "s_barrier\n\t" \
                 "s_cbranch_vccz .Lxret\n\t" \
                 "s_and_saveexec_b64 s[58:59], vcc\n\t" \
                 "global_store_dword v10, v11, s[64:65] sc0 sc1\n\t" \
                 "s_mov_b64 exec, s[58:59]\n\t" \
                 "s_branch .Lxret\n" \
                 ".Lxover:" \
                 :: [par] "i"(XSTAGE_PAR), [flag] "i"(2 * XSTAGE_PAR + (MS - XROWS) * KD), [n] "i"(NXS / 2 - 1))
static_assert(WS_CNT == 0 && NXS % 2 == 0 && XSTAGE_PAR < (1 << 24) && (MS - XROWS) * KD % 0x8000 == 0, "x send");

__global__ void __launch_bounds__(NTHR, 2) gemm_a2a(Params p) {
    __shared__ __attribute__((aligned(1024))) char smem_[NSLOT * SLOT_B + 16384 + 2048];
    LDSP char* smem = (LDSP char*)smem_;
    typedef unsigned u32x8 __attribute__((ext_vector_type(8)));
    typedef unsigned u32x2 __attribute__((ext_vector_type(2)));
    u32x8 hot, hot2, lnk;
    u32x2 pg01;
    unsigned tile1;
    u64 tstart;
    asm volatile("s_load_dwordx8 %0, %6, 0x0\n\ts_load_dwordx2 %1, %6, %7 offset:%9\n\ts_load_dwordx8 %5, %6, %8 offset:%11\n\t"
                 "s_load_dword %2, %6, %7 offset:%10\n\ts_load_dwordx8 %4, %6, 0x20\n\ts_memrealtime %3"
                 : "=&{s[20:27]}"(hot), "=&s"(pg01), "=&s"(tile1), "=&s"(tstart), "=&s"(hot2), "=&{s[64:71]}"(lnk)
                 : "s"(__builtin_amdgcn_kernarg_segment_ptr()), "s"(blockIdx.x * (unsigned)sizeof(Prog)), "s"((blockIdx.x & 7) * (unsigned)sizeof(Link)),
                   "i"(offsetof(Params, prog)), "i"(offsetof(Params, prog) + 8), "i"(offsetof(Params, link)));
    {
        u64 pc;
        unsigned o;
        asm volatile("s_cmp_lt_u32 %2, %4\n\ts_cbranch_scc1 8f\n\ts_getpc_b64 %0\n\ts_mov_b32 m0, %6\n\t"
                     "v_lshlrev_b32 %1, 7, %3\n\tv_min_u32 %1, %5, %1\n\tglobal_load_lds_dword %1, %0\n8:"
                     : "=&s"(pc), "=&v"(o) : "s"(blockIdx.x), "v"(threadIdx.x), "i"(GRID - 8), "i"(CODE_BYTES), "i"(NSLOT * SLOT_B + 16384)
                     : "memory", "scc");
    }
    __builtin_amdgcn_sched_barrier(0);
    const int tid = threadIdx.x, lane = tid & 63;
    const int wv = __builtin_amdgcn_readfirstlane(tid >> 6);
    Stream ks;
    {
        const int lw = wv & 3, lr = lane >> 3, sl = lane & 7;
        ks.ldsw = (unsigned)(size_t)smem + lw * 4096;
        ks.ldsx = BN * BK - lw * 2048;
        ks.w0 = (32 * lw + lr) * ND + ((sl ^ (lr >> 1)) << 4);
        ks.w1 = ks.w0 + 8 * ND - 1024; ks.w2 = (ks.w0 ^ 64) + 16 * ND - 2048; ks.w3 = ks.w2 + 8 * ND - 1024;
        ks.x0 = (8 * lw + lr) * KD + ((sl ^ ((lr >> 1) | ((lw & 1) << 2))) << 4);
        ks.x1 = ks.x0 + 32 * KD - 1024;
        asm volatile("" : "+v"(ks.w0), "+v"(ks.w1), "+v"(ks.w2), "+v"(ks.w3), "+v"(ks.x0), "+v"(ks.x1));
        asm volatile("" : "+s"(ks.ldsw), "+s"(ks.ldsx));
    }
    __builtin_amdgcn_sched_barrier(0);
    XHEAD();
    asm volatile("s_waitcnt lgkmcnt(0)" : "+s"(hot), "+s"(pg01), "+s"(tile1), "+s"(tstart), "+s"(hot2), "+s"(lnk));
    const char* const px = (const char*)((u64)hot[1] << 32 | hot[0]);
    const char* const pw = (const char*)((u64)hot[3] << 32 | hot[2]);
    char* const myws = (char*)(GLBP char*)((u64)hot[5] << 32 | hot[4]);
    const int rank = hot[6];
    char* const myout = (char*)(GLBP char*)((u64)hot2[5] << 32 | hot2[4]);
    const unsigned pre = pg01[0], tile0 = pg01[1];
    const bool early = wv >= 4 && (pre & 3) == PRE_NONE;
    if (early) stream_start(ks, pw + (tile0 >> 27) * BN, px + ((tile0 >> 3) & 31) * 16 * KD, 64 * KD, 0, tile0 & 4);
    __builtin_amdgcn_sched_barrier(0);
    unsigned* const tmo = (unsigned*)(myws + WS_TMO);
    unsigned* const mycnt = (unsigned*)(myws + WS_CNT) + blockIdx.x * 8 + wv;

    if ((pre & 3) != PRE_NONE) {
        if ((pre & 3) == PRE_XSEND) {
            XBODY();
            STAMP(4);
            if (wv >= 4) while (__builtin_amdgcn_s_memrealtime() - tstart < ((pre >> 2) & 2047)) __builtin_amdgcn_s_sleep(2);
        } else {
            const unsigned ticks = (pre >> 2) & 2047;
            const int job = (pre >> 13) & 31;
            const unsigned dmy = (unsigned)(size_t)smem + NSLOT * SLOT_B + 16384;
            if (job && job < 31) {
                const char* wp = pw + (tile0 >> 29) * NS_COLS;
                auto pfa = [&](int idx) {
                    const int st = idx >> 3, part = idx & 3;
                    if (idx & 4) touch(dmy, (64 * part + lane) * KD + st * BK, px);
                    else         touch(dmy, (st * BK + 32 * part + (lane >> 1)) * ND + (2 + (lane & 1)) * BN, wp);
                };
                auto pfb = [&](int idx) { touch(dmy, (256 + 64 * (idx % 3) + lane) * KD + (idx / 3) * BK, px); };
                pfa(wv * 16 + job - 1);
                if (job > 6) {
                    asm volatile("s_waitcnt vmcnt(0)" ::: "memory");
                    pfa(128 + wv * 10 + job - 7);
                    if (job > 8)  { asm volatile("s_waitcnt vmcnt(0)" ::: "memory"); pfb(wv * 8 + job - 9); }
                    if (job > 10) { asm volatile("s_waitcnt vmcnt(0)" ::: "memory"); pfa(208 + wv * 6 + job - 11); }
                    if (job > 12) { asm volatile("s_waitcnt vmcnt(0)" ::: "memory"); pfb(64 + wv * 4 + job - 13); }
                    STAMP(5);
                }
            }
            if (job == 31) {
                const int id = wv * 32 + (blockIdx.x >> 3);
                while (__builtin_amdgcn_s_memrealtime() - tstart < PF7T) __builtin_amdgcn_s_sleep(2);
#pragma unroll 1
                for (int idx = id; idx < 896; idx += 256) {
                    const int blk = idx >= 448, r = idx - 448 * blk;
                    touch(dmy, ((r / 7) * 64 + lane) * ND + blk * BN, pw + ((rank + 1 + r % 7) & (NR - 1)) * NS_COLS);
                    asm volatile("s_waitcnt vmcnt(0)" ::: "memory");
                }
                touch(dmy, (id * 16 + (lane >> 2)) * ND + (lane & 3) * BN, pw + rank * NS_COLS);
                STAMP(5);
            }
            if (wv >= 4) while (__builtin_amdgcn_s_memrealtime() - tstart < ticks) __builtin_amdgcn_s_sleep(2);
        }
    }

    unsigned call = 0;
    float scale0 = 0.f;
    auto fetch = [&]() {
        float sx, sw;
        asm volatile("s_load_dword %0, %3, 0x0\n\ts_load_dword %1, %4, 0x0\n\ts_load_dword %2, %5, 0x0\n\ts_waitcnt lgkmcnt(0)"
                     : "=&s"(call), "=&s"(sx), "=&s"(sw) : "s"(mycnt), "s"((u64)hot2[1] << 32 | hot2[0]), "s"((u64)hot2[3] << 32 | hot2[2]) : "memory");
        call += 1;
        scale0 = sx * sw;
    };
    if (!early) fetch();

    if (wv >= 4) {
        const int lw = wv - 4;
#pragma unroll 1
        for (int ti = 0; ti < 2; ++ti) {
            const TileInfo d = decode(ti ? tile1 : tile0);
            if (d.kind == T_NONE) break;
            const char* xbase = px + (size_t)d.row * 16 * KD;
            const char* wbase = pw + ((ti ? tile1 : tile0) >> 27) * BN;
            if (ti) asm volatile("s_barrier" ::: "memory");
            if (__builtin_expect(d.kind == T_PEER, 0)) {
                xbase = myws + WS_XSTAGE + (call & 1) * XSTAGE_PAR + (size_t)d.row * 64 * KD;
                asm volatile("s_barrier" ::: "memory");
            }
            const unsigned xbytes = d.nr * 16 * KD;
            if (ti || !early) stream_start(ks, wbase, xbase, xbytes, d.rot, d.mt - 1);
            if (d.mt == 2) stream_rest<2>(ks, wbase, xbase, xbytes, d.rot);
            else           stream_rest<1>(ks, wbase, xbase, xbytes, d.rot);
            if (!ti && early) fetch();
        }
        if (lane == 0) __hip_atomic_store(mycnt, call, __ATOMIC_RELAXED, __HIP_MEMORY_SCOPE_SYSTEM);
    } else {
        LDSP char* ep = smem + NSLOT * SLOT_B + wv * 4096;
        LDSP char* pp = smem + NSLOT * SLOT_B + 16384 + 256 + wv * 256;
        LDSP char* el = smem + NSLOT * SLOT_B + 16384 + 1536 + wv * 64;
        volatile LDSP unsigned* sig = (volatile LDSP unsigned*)(smem + NSLOT * SLOT_B + 16384 + 1280) + wv;
        *sig = 0;
#pragma unroll 1
        for (int ti = 0; ti < 2; ++ti) {
            const TileInfo d = decode(ti ? tile1 : tile0);
            if (d.kind == T_NONE) break;
            unsigned tmv = 0;
            int eb = 8, npk = d.pk, mt = d.mt;
            const int band = unit_band(mt, wv, 0);
            int nu = d.nr - band;
            nu = nu < 0 ? 0 : nu > mt ? mt : nu;
            int g0 = rank * (MS / 16) + d.row + band, g1 = g0 + 1;
            if (__builtin_expect(d.kind == T_PEER, 0)) {
                constexpr int XB = XROWS / 16;
                if (tid < 2 * NXPART) {
                    const int b = (4 * d.row + (tid < NXPART ? 0 : d.nr - 1)) / XB;
                    wait_flag((unsigned*)(myws + WS_XFLAG + b * XSTAGE_SRC) + (tid & (NXPART - 1)), call, tmo);
                }
                asm volatile("s_barrier" ::: "memory");
                tmv = __hip_atomic_load(tmo, __ATOMIC_RELAXED, __HIP_MEMORY_SCOPE_AGENT);
                const int r0 = 4 * d.row + band, r1 = r0 + 1;
                g0 = ((rank - 1 - r0 / XB) & (NR - 1)) * (MS / 16) + MS / 16 - XB + r0 % XB;
                g1 = ((rank - 1 - r1 / XB) & (NR - 1)) * (MS / 16) + MS / 16 - XB + r1 % XB;
            }
            char* const oc = (d.kind == T_REMOTE ? (char*)((u64)lnk[7] << 32 | lnk[6]) : myout) + unit_col(mt, d.y, wv);
            char* ob[2] = {oc + (unsigned)(g0 * 16 * NS_COLS * 4), oc + (unsigned)(g1 * 16 * NS_COLS * 4)};
            unsigned tsend = (unsigned)tstart + (pre >> 18);
            GLBP u64* link = (GLBP u64*)(myws + WS_LINK + d.bb * 128);
            int pull = d.pull;
            char* us = (pull ? myws + WS_USTAGE + PULL_OFF + ((rank + 1 + d.bb) & (NR - 1)) * USTAGE_SRC : (char*)((u64)lnk[3] << 32 | lnk[2]))
                       + (unsigned)((call & 1) * USTAGE_PAR + d.flag * UTILE + wv * mt * 3200 + lane * 16);
            char* fl0 = (char*)((u64)lnk[5] << 32 | lnk[4]) + (unsigned)((call & 1) * UFLAG_PAR + (d.flag * 4 + wv) * 64);
            asm volatile("" : "+v"(us));
            asm volatile("" : "+s"(fl0), "+s"(ob[0]), "+s"(ob[1]), "+s"(npk), "+s"(nu), "+s"(tsend), "+s"(pull));
            f32x4 v[8];
            unsigned kp = 0, tnow = 0;
            if (mt == 1) compute<1>(smem, wv, lane, v, eb, tnow, kp);
            else         compute<2>(smem, wv, lane, v, eb, tnow, kp);
            tnow += 55 + 35 * mt;
            asm volatile("" : "+v"(tmv));
            const float scale = tmv == call ? __builtin_nanf("") : scale0;
            const float sfw = __builtin_bit_cast(float, (254 - eb) << 23), sinv = __builtin_bit_cast(float, eb << 23);
            PROF(0);
            u32x4 dat[2][4];
            asm volatile("" : "=v"(dat[0][0]), "=v"(dat[0][1]), "=v"(dat[0][2]), "=v"(dat[0][3]),
                              "=v"(dat[1][0]), "=v"(dat[1][1]), "=v"(dat[1][2]), "=v"(dat[1][3]));
            bool rawu[2] = {true, true}, pend[2] = {nu > 0, nu > 1};
            unsigned off = 0, step = 0;
            auto send_unit = [&](int h) {
                if (!rawu[h]) {
#pragma unroll
                    for (int u = 0; u < 3; ++u) st_wt(us + h * 3072 + u * 1024, dat[h][u]);
                } else {
#pragma unroll
                    for (int u = 0; u < 4; ++u) st_wt(ob[h] + off + u * step, dat[h][u]);
                }
                pend[h] = false;
            };
            auto due = [&](unsigned now) { return now + DUE - tsend < 0x80000000u; };
            auto make_raw = [&](int h) {
                int ln = lane;
                asm volatile("" : "+v"(ln));
                piece_geom(mt, ln, off, step);
#pragma unroll
                for (int f = 0; f < 4; ++f) *(LDSP f32x4*)(ep + frag_slot(mt, ln, f)) = v[4 * h + f] * scale;
                asm volatile("" ::: "memory");
#pragma unroll
                for (int u = 0; u < 4; ++u) dat[h][u] = *(LDSP u32x4*)(ep + piece_slot(mt, ln, u));
                asm volatile("" ::: "memory");
                rawu[h] = true;
            };
            const unsigned tsend0 = tsend;
            const bool late = due(tnow) || pull;
            u64 slip = 0;
            if (!late && mt == 2) slip = __hip_atomic_load(link, __ATOMIC_RELAXED, __HIP_MEMORY_SCOPE_AGENT);
            auto reslot = [&]() {
                tsend = tsend0 + (__builtin_amdgcn_readfirstlane((unsigned)(slip >> 32)) == call ? __builtin_amdgcn_readfirstlane((unsigned)slip) : 0);
            };
            unsigned hdr = 0xffff;
            if (__builtin_expect(npk, 1)) {
                int q[2][16];
#pragma unroll
                for (int e = 0; e < 32; e += 8)
                    asm volatile("" : "=v"(q[e >> 4][e & 15]), "=v"(q[e >> 4][(e & 15) + 1]), "=v"(q[e >> 4][(e & 15) + 2]), "=v"(q[e >> 4][(e & 15) + 3]),
                                      "=v"(q[e >> 4][(e & 15) + 4]), "=v"(q[e >> 4][(e & 15) + 5]), "=v"(q[e >> 4][(e & 15) + 6]), "=v"(q[e >> 4][(e & 15) + 7]));
#pragma unroll
                for (int h = 0; h < 2; ++h) {
                    if (h >= nu) break;
                    unsigned dw[12];
                    f32x4 tf[4];
#pragma unroll
                    for (int f = 0; f < 4; ++f) tf[f] = v[4 * h + f] * sfw;
                    asm("" : "+v"(tf[0]), "+v"(tf[1]), "+v"(tf[2]), "+v"(tf[3]));
#pragma unroll
                    for (int f = 0; f < 4; ++f) {
#pragma unroll
                        for (int e = 0; e < 4; ++e) { const float te = tf[f][e]; asm("v_cvt_i32_f32 %0, %1" : "=v"(q[h][4 * f + e]) : "v"(te)); }
                        dw[3 * f]     = __builtin_amdgcn_perm(q[h][4 * f + 1], q[h][4 * f], 0x04020100);
                        dw[3 * f + 1] = __builtin_amdgcn_perm(q[h][4 * f + 2], q[h][4 * f + 1], 0x05040201);
                        dw[3 * f + 2] = __builtin_amdgcn_perm(q[h][4 * f + 3], q[h][4 * f + 2], 0x06050402);
                    }
#pragma unroll
                    for (int u = 0; u < 3; ++u) dat[h][u] = (u32x4){dw[4 * u], dw[4 * u + 1], dw[4 * u + 2], dw[4 * u + 3]};
                    rawu[h] = false;
                    if (!h) PROF(1);
                    if (late) send_unit(h);
                }
                auto send_if_near = [&](unsigned now) {
                    if (now + NEAR - tsend < 0x80000000u) {
                        while ((unsigned)__builtin_amdgcn_s_memrealtime() - tsend > 0x80000000u) __builtin_amdgcn_s_sleep(1);
                        if (pend[0]) send_unit(0);
                        if (pend[1]) send_unit(1);
                    }
                };
                if (!late) {
                    __builtin_amdgcn_sched_barrier(0);
                    reslot();
                    send_if_near((unsigned)__builtin_amdgcn_s_memrealtime());
                }
                const int by = (int)(tnow + SLIPADD - tsend0);
                if (by > 0 && !pull && tid == 0) __hip_atomic_fetch_max(link, (u64)call << 32 | (unsigned)(by < 200 ? by : 200), __ATOMIC_RELAXED, __HIP_MEMORY_SCOPE_AGENT);
                PROF(2);
                int ln = lane, nesc = 0;
                asm volatile("" : "+v"(ln));
                if (ln < 8) *(LDSP u64*)(el + ln * 8) = (u64)(call << 13) << 32;
#pragma unroll
                for (int h = 0; h < 2; ++h) {
                    if (h >= nu) break;
                    const unsigned now = (unsigned)__builtin_amdgcn_s_memrealtime();
                    unsigned pa = 0, pb = 0;
#pragma unroll
                    for (int e = 0; e < 8; ++e) { pa = __builtin_amdgcn_alignbit(pa, q[h][e], 31); pb = __builtin_amdgcn_alignbit(pb, q[h][8 + e], 31); }
                    *(LDSP unsigned short*)(pp + h * 128 + ln * 2) = (unsigned short)(pa << 8 | pb);
                    unsigned mm[2] = {0, 0};
                    float val[2] = {0.f, 0.f};
#pragma unroll
                    for (int f = 0; f < 4; ++f) {
                        f32x4 rc;
#pragma unroll
                        for (int e = 0; e < 4; ++e) rc[e] = __builtin_amdgcn_fmed3f((float)q[h][4 * f + e], -16777216.f, 16777215.f);
                        rc *= sinv;
                        const float x0 = v[4 * h + f][0], x1 = v[4 * h + f][1], x2 = v[4 * h + f][2], x3 = v[4 * h + f][3];
                        const float r0 = rc[0], r1 = rc[1], r2 = rc[2], r3 = rc[3];
                        u64 c0, c1, c2, c3;
                        asm("v_cmp_ne_u32 %2, %6, %10\n\tv_cmp_ne_u32 %3, %7, %11\n\tv_cmp_ne_u32 %4, %8, %12\n\tv_cmp_ne_u32 %5, %9, %13\n\t"
                            "v_cndmask_b32 %1, %1, %6, %2\n\tv_addc_co_u32 %0, vcc, %0, %0, %2\n\t"
                            "v_cndmask_b32 %1, %1, %7, %3\n\tv_addc_co_u32 %0, vcc, %0, %0, %3\n\t"
                            "v_cndmask_b32 %1, %1, %8, %4\n\tv_addc_co_u32 %0, vcc, %0, %0, %4\n\t"
                            "v_cndmask_b32 %1, %1, %9, %5\n\tv_addc_co_u32 %0, vcc, %0, %0, %5"
                            : "+v"(mm[f >> 1]), "+v"(val[f >> 1]), "=&s"(c0), "=&s"(c1), "=&s"(c2), "=&s"(c3)
                            : "v"(x0), "v"(x1), "v"(x2), "v"(x3), "v"(r0), "v"(r1), "v"(r2), "v"(r3) : "vcc");
                    }
                    const unsigned m16 = mm[0] << 8 | mm[1];
                    const u64 has = __builtin_amdgcn_ballot_w64(m16 != 0);
                    const int e = __builtin_clz(m16 | 1) - 16;
                    const int slot = nesc + 1 + __builtin_amdgcn_mbcnt_hi((unsigned)(has >> 32), __builtin_amdgcn_mbcnt_lo((unsigned)has, 0));
                    const float x = mm[0] ? val[0] : val[1];
                    if (m16 && slot <= NESC)
                        *(LDSP u64*)(el + slot * 8) = (u64)(call << 13 | h << 12 | (frag_slot(mt, ln, e >> 2) + 4 * (e & 3)) | 1) << 32 | __builtin_bit_cast(unsigned, x);
                    const int n = nesc + __builtin_popcountll(has);
                    if (__builtin_expect(n > NESC || __builtin_amdgcn_ballot_w64((m16 & (m16 - 1)) != 0), 0)) {
                        make_raw(h);
                        pend[h] = true;
                    } else {
                        nesc = n;
                        hdr ^= (255u ^ (unsigned)eb) << (8 * h);
                    }
                    if (!h && nu > 1 && pend[1]) send_if_near(now + 45);
                    if (!h) PROF(3);
                }
            } else {
                if (lane < 8) *(LDSP u64*)(el + lane * 8) = (u64)(call << 13) << 32;
#pragma unroll
                for (int h = 0; h < 2; ++h) {
                    if (h >= nu) break;
                    make_raw(h);
                    if (!h && late) send_unit(0);
                }
                if (!late) reslot();
            }
            asm volatile("" ::: "memory");
            const u32x4 pv = *(LDSP u32x4*)(pp + (lane & 15) * 16);
            u64 fv = *(LDSP u64*)(el + (lane & 7) * 8);
            if (lane == 0) fv = (u64)call << 32 | hdr;
            char* fa = fl0 + lane * 8;
            asm volatile("" : "+v"(fa), "+v"(fv));
            PROF(4);
            if (pull) {
                if (lane < 8 * nu) st_wt(us + mt * 3072, pv);
                slip = __hip_atomic_load(link, __ATOMIC_RELAXED, __HIP_MEMORY_SCOPE_AGENT);
                reslot();
            } else if (pend[0] && (unsigned)__builtin_amdgcn_s_memrealtime() + 70 - tsend > 0x80000000u) {
                while ((unsigned)__builtin_amdgcn_s_memrealtime() + 70 - tsend > 0x80000000u) __builtin_amdgcn_s_sleep(2);
                slip = __hip_atomic_load(link, __ATOMIC_RELAXED, __HIP_MEMORY_SCOPE_AGENT);
                reslot();
            }
            while ((unsigned)__builtin_amdgcn_s_memrealtime() - tsend > 0x80000000u) __builtin_amdgcn_s_sleep(1);
            PROF(5);
            if (pend[0]) send_unit(0);
            if (pend[1]) send_unit(1);
            if (npk && !pull && lane < 8 * nu) st_wt(us + mt * 3072, pv);
            if (ti == 0 && (tile1 & 3) != T_NONE) asm volatile("s_barrier" ::: "memory");
            PROF(6);
            const u64 who = d.kind == T_REMOTE && nu ? 0xff : 0;
            asm volatile("s_mov_b64 exec, %3\n\ts_waitcnt vmcnt(0)\n\ts_cbranch_execz 1f\n\t"
                         "global_store_dwordx2 %1, %2, off sc0 sc1\n1:\n\ts_mov_b64 exec, -1"
                         :: "v"(slip), "v"(fa), "v"(fv), "s"(who) : "memory");
            *sig = 1;
            if (d.kind == T_REMOTE && nu) {
                PROF(7);
            }
            if (!ti && lane == 0) __hip_atomic_store(mycnt, call, __ATOMIC_RELAXED, __HIP_MEMORY_SCOPE_SYSTEM);
            if (ti == 0) STAMP(3); else STAMP(7);
            if (ti == 0) PROFOUT(0, 8);
        }
    }

#pragma unroll 1
    for (int ri = 0; ri < 2; ++ri) {
        unsigned tw = ri ? tile0 : tile1;
        asm volatile("" : "+s"(tw));
        const TileInfo d = decode(tw);
        const int h = wv < 4, wq = wv & 3;
        if (d.own && (d.kind == T_REMOTE) == ri && h < d.mt && unit_band(d.mt, wq, h) < d.nr && (!h || d.pk)) {
            int ln = lane;
            asm volatile("" : "+v"(ln));
            const int src = (rank + 1 + d.bb) & (NR - 1);
            float sc = scale0;
            unsigned off, step;
            piece_geom(d.mt, ln, off, step);
            char* o = myout + (size_t)(src * (MS / 16) + d.row + unit_band(d.mt, wq, h)) * 16 * NS_COLS * 4 + unit_col(d.mt, d.y, wq) + off;
            char* o2 = o + 2 * step;
            const char* us = (d.pull ? (char*)((u64)lnk[3] << 32 | lnk[2]) + PULL_OFF : myws + WS_USTAGE + src * USTAGE_SRC)
                             + (unsigned)((call & 1) * USTAGE_PAR + d.flag * UTILE + wq * d.mt * 3200);
            const u64* fl = (const u64*)(myws + WS_UFLAG + (call & 1) * UFLAG_PAR + src * UFLAG_SRC) + (d.flag * 4 + wq) * 8 + (ln & 7);
            const char* a0 = us + h * 3072 + ln * 16;
            const char* a1 = us + d.mt * 3072 + h * 128 + ln * 2;
            const unsigned rp = RP_SLOT * SLOT_B + wv * 4096;
            unsigned fs[4], ps[2];
#pragma unroll
            for (int f = 0; f < 4; ++f) fs[f] = rp + frag_slot(d.mt, ln, f);
            ps[0] = rp + piece_slot(d.mt, ln, 0); ps[1] = rp + piece_slot(d.mt, ln, 1);
            char* o1 = o + step;
            char* o3 = o2 + step;
            asm volatile("" : "+v"(a0), "+v"(a1), "+v"(o), "+v"(o1), "+v"(o2), "+v"(o3), "+v"(sc), "+v"(fl),
                              "+v"(fs[0]), "+v"(fs[1]), "+v"(fs[2]), "+v"(fs[3]), "+v"(ps[0]), "+v"(ps[1]));
            u64 fw;
            unsigned key = ln & 7 ? call << 13 : call, lim = ln & 7 ? 0x1fff : 0;
            asm volatile("" : "+v"(key), "+v"(lim));
            volatile LDSP unsigned* sig = (volatile LDSP unsigned*)(smem + NSLOT * SLOT_B + 16384 + 1280) + wq;
            if (!ri) while ((unsigned)__builtin_amdgcn_s_memrealtime() - (unsigned)tstart < TFIRST + 260) __builtin_amdgcn_s_sleep(4);
            else if (!h) {
                while (__builtin_amdgcn_readfirstlane(*sig) == 0) __builtin_amdgcn_s_sleep(4);
                __builtin_amdgcn_s_sleep(PSLEEP);
            }
            PROF(0);
            const bool ok = wait_uflag(fl, key, lim, __builtin_amdgcn_readfirstlane(ri & ~d.pk), call, tmo, fw);
            PROF(1);
            const unsigned hb = (__builtin_amdgcn_readfirstlane((unsigned)fw) >> (8 * h)) & 255;
            if (__builtin_expect(!ok, 0)) {
                st_wt1(o, __builtin_nanf(""));
                st_wt1(myout + (size_t)rank * MS * NS_COLS * 4, __builtin_nanf(""));
            }
            else if (hb != 255) {
                u32x4 a, b, c;
                unsigned pl;
                asm volatile("global_load_dwordx4 %0, %4, off sc0 sc1\n\t"
                             "global_load_dwordx4 %1, %4, off offset:1024 sc0 sc1\n\t"
                             "global_load_dwordx4 %2, %4, off offset:2048 sc0 sc1\n\t"
                             "global_load_ushort %3, %5, off sc0 sc1\n\ts_waitcnt vmcnt(0)"
                             : "=&v"(a), "=&v"(b), "=&v"(c), "=&v"(pl) : "v"(a0), "v"(a1) : "memory");
                PROF(2);
                const unsigned dw[12] = {a.x, a.y, a.z, a.w, b.x, b.y, b.z, b.w, c.x, c.y, c.z, c.w};
                const float sinv = __builtin_bit_cast(float, (hb - 7) << 23);
                u32x4 t[4], sg[4];
#pragma unroll
                for (int f = 0; f < 4; ++f) {
                    t[f] = (u32x4){dw[3 * f] << 8, __builtin_amdgcn_perm(dw[3 * f + 1], dw[3 * f], 0x0504030c),
                                   __builtin_amdgcn_perm(dw[3 * f + 2], dw[3 * f + 1], 0x0403020c), dw[3 * f + 2] & 0xffffff00u};
                    sg[f] = (u32x4){pl >> (15 - 4 * f), pl >> (14 - 4 * f), pl >> (13 - 4 * f), pl >> (12 - 4 * f)};
                }
                asm volatile("" : "+v"(t[0]), "+v"(t[1]), "+v"(t[2]), "+v"(t[3]), "+v"(sg[0]), "+v"(sg[1]), "+v"(sg[2]), "+v"(sg[3]));
#pragma unroll
                for (int f = 0; f < 4; ++f) {
                    const f32x4 r = {(float)(int)__builtin_amdgcn_alignbit(sg[f].x, t[f].x, 1), (float)(int)__builtin_amdgcn_alignbit(sg[f].y, t[f].y, 1),
                                     (float)(int)__builtin_amdgcn_alignbit(sg[f].z, t[f].z, 1), (float)(int)__builtin_amdgcn_alignbit(sg[f].w, t[f].w, 1)};
                    *(LDSP f32x4*)(smem + fs[f]) = r * sinv;
                }
                const unsigned where = (unsigned)(fw >> 32);
                asm volatile("" ::: "memory");
                if (ln && ln <= NESC && (where & 0x1001) == (unsigned)(h << 12 | 1)) *(LDSP unsigned*)(smem + rp + (where & 0xffc)) = (unsigned)fw;
                asm volatile("" ::: "memory");
                f32x4 r[4];
#pragma unroll
                for (int u = 0; u < 4; ++u) r[u] = *(LDSP f32x4*)(smem + ps[u & 1] + (u >> 1) * 2048);
                char* const op[4] = {o, o1, o2, o3};
#pragma unroll
                for (int u = 0; u < 4; ++u) st_wt(op[u], __builtin_bit_cast(u32x4, r[u] * sc));
            }
            PROF(4);
        }
    }
}

#define PREF 1
#define DELAYL 400
#define TCAP 620
#define TCAPS 35
#define STAG0 0
#define STAG1 0
#define TLEAD 1000
#define SROT 4
static unsigned pack_pre(int kind, int ticks, int job, int send) { return kind | ticks << 2 | job << 13 | (unsigned)send << 18; }
static void make_plan(Params& p, void* const* d_out_all, void* const* d_ws_all) {
    static const unsigned char near03[NR] = {0x26, 0x2c, 0x29, 0x54, 0x15, 0x4a, 0x1a, 0x32};
    int role[8];
    for (int x = 0, a = 0, b = 0; x < 8; ++x) {
        if (x == XLOC) { role[x] = 7; continue; }
        int& c = x < 4 ? a : b;
        while (((near03[p.rank] >> c) & 1) != (x < 4)) ++c;
        role[x] = c++;
    }
    for (int x = 0; x < 8; ++x) {
        const int peer = (p.rank + 1 + role[x]) % NR;
        char* const ws = (char*)d_ws_all[peer];
        p.link[x] = {ws + WS_XSTAGE + role[x] * XSTAGE_SRC - (size_t)(MS - XROWS) * KD, ws + WS_USTAGE + p.rank * USTAGE_SRC,
                     ws + WS_UFLAG + p.rank * UFLAG_SRC, (char*)d_out_all[peer]};
    }
    constexpr int RB = (MS - XROWS) / 16;
    constexpr int PB = 7 * XROWS / 16, PG = (PB + 3) / 4;
    static_assert(RB > 24 && RB <= 28 && PG <= 13, "plan");
    auto peer_tile = [&](int g, int y) { return pack_tile(p.rank, T_PEER, 2, g, 7, y, 0, g == PG - 1 ? PB - 4 * (PG - 1) : 4, 0, 0, 0); };
    unsigned local[32], second[56];
    constexpr int XG = PG > 8 ? PG - 8 : 0;
#define NLOC 5,5,11,11
#define NSEC 16,16,8,16
    const int nloc[4] = {NLOC}, nsec[4] = {NSEC};
    for (int y = 0, nl = 0, n0 = 0; y < 4; n0 += nsec[y++]) {
        int r = 32, g = 0;
        for (int j = 0; j < nloc[y]; ++j) {
            const unsigned rot = (unsigned)((32 * j / PG) & 31) << 13;
            if (g < PG) local[nl++] = peer_tile(g++, y) | rot;
            else { r -= 4; local[nl++] = pack_tile(p.rank, T_OWN, 2, r, 7, y, 0, 4, 0, 0, 0) | rot; }
        }
        int nbig = r / 2 - (nsec[y] - (PG - g));
        for (int pass = 0; pass < 8; ++pass)
            for (int n = n0; n < n0 + nsec[y]; ++n) {
                if ((n & 7) != pass) continue;
                if (g < PG) second[n] = peer_tile(g++, y);
                else if (r > 0) { const int mt = nbig-- > 0 ? 2 : 1; r -= 2 * mt; second[n] = pack_tile(p.rank, T_OWN, mt, r, 7, y, 0, 2 * mt, 0, 0, 0); }
                else second[n] = 0;
            }
        if (r != 0 || g < PG) fprintf(stderr, "plan: local tiles left over\n");
    }
    for (int b = 0; b < GRID; ++b) {
        const int x = role[b & 7], i = b >> 3;
        Prog& g = p.prog[b];
        g.pre = PRE_NONE; g.t[0] = g.t[1] = 0;
        if (x < 7) {
            const double up = PK ? 6.4 : 8.1, ur = 8.1;
            if (i >= 4 && i < 12) {
                g.pre = pack_pre(PRE_NONE, 0, 0, TFIRST + (int)((i - 4) * 2 * up * 100 / RATE));
                g.t[0] = pack_tile(p.rank, T_REMOTE, 1, 2 * ((i - 4) >> 1), x, 2 + (i & 1), 0, 2, i, PK, 0);
                g.t[1] = second[8 * x + i - 4] ? second[8 * x + i - 4] | (unsigned)(SROT * (i - 4)) << 13 : 0;
            } else {
                const int k = i < 4 ? 20 + i : i - 12;
                auto row_of = [](int k) { return k < 10 ? 8 + 4 * (k >> 1) : k < 20 ? 4 * ((k - 10) % 6) : k < 22 ? 24 : 4 * (k - 18); };
                auto y_of = [](int k) { return k < 10 ? 2 + (k & 1) : k < 20 ? (k - 10) / 6 : k != 20; };
                auto nr_of = [&](int k) { return RB - row_of(k) < 4 ? RB - row_of(k) : 4; };
                double before = 16 * up;
                for (int q = 0; q < k; ++q) before += nr_of(q) * (q >= 24 - NRAW ? ur : up);
                const int send = TFIRST + (int)(before * 100 / RATE);
                int start = k >= 10 ? send - TLEAD : k >= 4 ? STAG0 + (k - 4) * STAG1 : 0;
                if (start > TCAP) start = TCAP + (start - TCAP) * TCAPS / 100;
                const int pk = k >= 24 - NRAW ? 0 : PK, pull = pk && k >= 24 - NPULL;
                const int t = pull ? send - PLEAD : send;
                if (i < 4)       g.pre = pack_pre(PRE_XSEND, start, i, t);
                else if (k >= 4) g.pre = pack_pre(PRE_DELAY, start, PREF ? k - 3 : 0, t);
                else             g.pre = pack_pre(PRE_NONE, 0, 0, t);
                g.t[0] = pack_tile(p.rank, T_REMOTE, nr_of(k) == 2 ? 1 : 2, row_of(k), x, y_of(k), pull, nr_of(k), i, pk, 1);
                if (k >= 2 && k < 10)
                    g.t[1] = pack_tile(p.rank, T_NONE, 1, 2 * ((k - 2) >> 1), x, 2 + (k & 1), 0, 2, k + 2, PK, 1);
            }
        } else {
            g.pre = pack_pre(PRE_DELAY, DELAYL, 31, 0);
            g.t[0] = local[i];
        }
    }
}

extern "C" void kernel_launch(int rank, int nranks, ncclComm_t comm,
                              hipStream_t stream,
                              void* const* d_in, const int* in_sizes, int n_in,
                              void* d_out, int out_size,
                              const int* in_dtypes, int out_dtype,
                              void* const* d_out_all,
                              void* const* d_ws_all, size_t ws_bytes) {
    if (nranks != NR || (unsigned)rank >= (unsigned)NR || n_in < 4 || ws_bytes < WS_NEED ||
        in_sizes[0] != MS * KD || in_sizes[1] != KD * ND || out_size != NR * MS * NS_COLS) {
        fprintf(stderr, "kernel_launch: unexpected problem shape\n");
        return;
    }
    Params p;
    p.x = (const char*)d_in[0];
    p.w = (const char*)d_in[1];
    p.sx = (const float*)d_in[2];
    p.sw = (const float*)d_in[3];
    p.ws = (char*)d_ws_all[rank];
    p.out = (char*)d_out;
    p.rank = rank;
    p.pad = 0;
    p.pad2 = nullptr;
    make_plan(p, d_out_all, d_ws_all);
    gemm_a2a<<<GRID, NTHR, 0, stream>>>(p);
}
```
